```python
import numpy as np

kernel_name = "gpu_gemm_ar_m4096_k4096_n16384_e5m2e5m2_dqsilu"


ABSMAX_RTOL = 0.05

COMPUTE_DTYPE = "float32"
EPILOGUE = "dequant_silu"
IN_QUANT = None
OUT_QUANT = None
IN_FP8 = "e5m2e5m2"
IN_DTYPES = {"x": "e5m2", "w_mat": "e5m2",
             "scale_x": "float32", "scale_w": "float32"}
WORLD_SIZE = 8

def _snap_fp8(x, mant_bits, max_abs, min_normal):
    a = np.abs(x.astype(np.float64))
    _m, _e = np.frexp(a)
    q = np.where(a >= min_normal, np.ldexp(1.0, _e - 1 - mant_bits),
                 min_normal * 2.0 ** -mant_bits)
    snapped = np.minimum(np.round(a / q) * q, max_abs)
    return (np.sign(x) * snapped).astype(np.float32)

def setup_inputs():

    x = _snap_fp8(np.random.randn(4096, 4096).astype(np.float32), 2, 57344.0, 2.0 ** -14)
    w_mat = _snap_fp8(np.random.randn(4096, 16384).astype(np.float32), 2, 57344.0, 2.0 ** -14)
    scale_x = (0.02 * (1.0 + np.random.rand(1))).astype(np.float32)
    scale_w = (0.02 * (1.0 + np.random.rand(1))).astype(np.float32)
    return {"x": x, "w_mat": w_mat, "scale_x": scale_x, "scale_w": scale_w}

def reference(x, w_mat, scale_x, scale_w):
    acc = (x @ w_mat).astype(np.float32)
    y = (acc * (scale_x * scale_w)).astype(np.float32)
    return (y / (1.0 + np.exp(-np.clip(y, -60.0, 60.0)))).astype(np.float32)
```

```cpp
#include <hip/hip_runtime.h>
#include <rccl/rccl.h>
#include <cstdio>

#define HIP_CHECK(expr) do { hipError_t _e = (expr); if (_e != hipSuccess) { \
    fprintf(stderr, "HIP error %s at %s:%d\n", hipGetErrorString(_e), __FILE__, __LINE__); \
    return; } } while (0)

namespace {

#define LAS __attribute__((address_space(3)))
typedef int i32x4 __attribute__((ext_vector_type(4)));
typedef int i32x8 __attribute__((ext_vector_type(8)));
typedef unsigned u32x4 __attribute__((ext_vector_type(4)));
typedef float f32x4 __attribute__((ext_vector_type(4)));
typedef float f32x2 __attribute__((ext_vector_type(2)));

constexpr int M_ = 4096, N_ = 16384, NR = 8, KSH = 512;
constexpr int NWG = 256, NTHR = 512;
constexpr unsigned UNIT = 32768;
constexpr unsigned HTB = 16384;
constexpr int XU = 64, NU = 320;
constexpr unsigned SRC_BYTES = NU * UNIT;
constexpr unsigned PAR_BYTES = NR * SRC_BYTES;
constexpr unsigned WS_CTR = 0;
constexpr unsigned WS_TMO = 16384;
constexpr unsigned WS_LF = 32768;
constexpr unsigned WS_SF = 65536;
constexpr unsigned WS_DBG = 512 * 1024;
constexpr unsigned WS_STAGE = 1u << 20;
constexpr size_t WS_NEED = (size_t)WS_STAGE + 2ull * PAR_BYTES;
constexpr unsigned long long DEADLINE_TICKS = 40000000ull;

#define NPX 4
constexpr int GX = 32 - NPX;
constexpr int G_ = 8 * GX;
constexpr int NPULL = 8 * NPX;
constexpr int CR = G_ / 16;
constexpr int E_ = 1024 - 4 * G_;
constexpr int EX = E_ / 8;
constexpr int C0 = 64 - 3 * CR;
constexpr int SR = (E_ == 0) ? 0 : (E_ == 64 ? 8 : 16);
constexpr int SC = (E_ == 0) ? 0 : E_ / (SR ? SR : 1);
static_assert(G_ % 16 == 0 && SR * SC == E_ && EX <= GX && SC <= C0, "bad split");

constexpr int LDS_BYTES = 8 * HTB + 2048;
#define PULL_ST "sc1"
#define ROT 1
#define PULLERS ((8 * NPX) / 7 * 7)
#define OUT_ST "sc1"
#define PULL_AUX 17

struct Params {
    const unsigned char* x; const unsigned char* w; const float* sx; const float* sw;
    float* out;
    unsigned char* ws[NR];
    int rank; int mode;
};

__device__ __forceinline__ int unit_of(int o, int q) {
    constexpr int nS = SR + SC, n0 = (16 - SR) + (C0 - SC);
    if (nS > 0 && o < 4 * nS) { const int j = o / (nS ? nS : 1); int i = o - j * nS + (q * nS) / 7; if (i >= nS) i -= nS;
        if (SR == SC) return (i & 1) ? XU + (i >> 1) * 4 + j : (i >> 1) * 4 + j;
        return i < SR ? i * 4 + j : XU + (i - SR) * 4 + j; }
    o -= 4 * nS;
    if (o < 4 * n0) { const int j = o / n0; int i = o - j * n0 + (q * n0) / 7; if (i >= n0) i -= n0;
        constexpr int nx = 16 - SR;
        if (i < 2 * nx) return (i & 1) ? XU + (SC + (i >> 1)) * 4 + j : (SR + (i >> 1)) * 4 + j;
        return XU + (SC + (i - nx)) * 4 + j; }
    o -= 4 * n0;
    const int r = o / (4 * CR), rem = o - r * 4 * CR, j = rem / CR; int i = rem - j * CR + (q * CR) / 7; if (i >= CR) i -= CR;
    return XU + (C0 + r * CR + i) * 4 + j;
}
static_assert((C0 - SC) >= (16 - SR), "unit_of assumes at least as many w as x panels in round 0");

__device__ __forceinline__ bool tile_of(int xcd, int slot, int i, int& pm, int& pn) {
    if (slot < EX) {
        if (i == 0) { const int idx = xcd * EX + slot, b = idx / (4 * (SC ? SC : 1)), rem = idx - b * 4 * SC; pn = rem >> 2; pm = 4 * b + (rem & 3); return true; }
        --i;
    }
    if (i > 3) return false;
    int idx = xcd * GX + slot;
    if (i == 0) {
        constexpr int nbS = SR / 4, a = 4 * (C0 - SC), c = 4 * C0;
        if (idx < nbS * a) { const int b = idx / a, rem = idx - b * a; pn = SC + (rem >> 2); pm = 4 * b + (rem & 3); }
        else { idx -= nbS * a; const int b = idx / c, rem = idx - b * c; pn = rem >> 2; pm = 4 * (nbS + b) + (rem & 3); }
        return true;
    }
    const int b = idx / (4 * CR), rem = idx - b * 4 * CR;
    pn = C0 + (i - 1) * CR + (rem >> 2); pm = 4 * b + (rem & 3);
    return true;
}

#define WAIT_V(n) asm volatile("s_waitcnt vmcnt(" #n ")" ::: "memory")
#define WAIT_L(n) asm volatile("s_waitcnt lgkmcnt(" #n ")" ::: "memory")
#define BAR __builtin_amdgcn_s_barrier()
#define SCHED __builtin_amdgcn_sched_barrier(0)
#define STAMP(i) do { } while (0)

__device__ __forceinline__ void st16_sys(void* p, u32x4 v) { asm volatile("global_store_dwordx4 %0, %1, off sc0 sc1\n\ts_nop 1" :: "v"(p), "v"(v) : "memory"); }
__device__ __forceinline__ void st16_dev(void* p, u32x4 v) { asm volatile("global_store_dwordx4 %0, %1, off sc1\n\ts_nop 1" :: "v"(p), "v"(v) : "memory"); }
__device__ __forceinline__ void ld4_dev_async(unsigned& v, const void* p) { asm volatile("global_load_dword %0, %1, off sc1" : "+v"(v) : "v"(p) : "memory"); }
__device__ __forceinline__ unsigned ld4_sys_now(const void* p) { unsigned v; asm volatile("global_load_dword %0, %1, off sc0 sc1\n\ts_waitcnt vmcnt(0)" : "=&v"(v) : "v"(p) : "memory"); return v; }
__device__ __forceinline__ unsigned ld4_dev_now(const void* p) { unsigned v; asm volatile("global_load_dword %0, %1, off sc1\n\ts_waitcnt vmcnt(0)" : "=&v"(v) : "v"(p) : "memory"); return v; }

template <bool SYS>
__device__ __noinline__ void wait_flags(const unsigned* flag, unsigned call, unsigned* tmo) {
    const unsigned long long t0 = __builtin_amdgcn_s_memrealtime();
    for (;;) {
        const unsigned v = SYS ? ld4_sys_now(flag) : ld4_dev_now(flag);
        if (__all(v >= call)) break;
        const unsigned t = ld4_dev_now(tmo);
        if (__builtin_amdgcn_readfirstlane(t) == call) break;
        if (__builtin_amdgcn_s_memrealtime() - t0 > DEADLINE_TICKS) {
            __hip_atomic_store(tmo, call, __ATOMIC_RELAXED, __HIP_MEMORY_SCOPE_AGENT);
            break;
        }
        __builtin_amdgcn_s_sleep(8);
    }
}

__device__ __forceinline__ unsigned lf_index(int u, int src) {
    return (unsigned)(u >> 2) * 32u + (unsigned)(src * 4 + (u & 3));
}

__device__ __noinline__ int pull_horizon(const LAS unsigned* tab, int horizon, int k, int nstep, const unsigned char* ws, unsigned call, unsigned* tmo, int lane) {
    const unsigned long long t0 = __builtin_amdgcn_s_memrealtime();
    while (horizon <= k) {
        const int i = horizon + lane < nstep ? horizon + lane : nstep - 1;
        const unsigned d = tab[i];
        const unsigned v = ld4_sys_now(ws + WS_SF + (((d >> 16) & (NR - 1)) * NU + (d & 0xffffu)) * 64u);
        const unsigned long long nr = ~__ballot(v >= call);
        const int cnt = nr ? __builtin_ctzll(nr) : 64;
        horizon += cnt;
        if (horizon > k) break;
        const unsigned t = ld4_dev_now(tmo);
        if (__builtin_amdgcn_readfirstlane(t) == call) return nstep;
        if (__builtin_amdgcn_s_memrealtime() - t0 > DEADLINE_TICKS) {
            __hip_atomic_store(tmo, call, __ATOMIC_RELAXED, __HIP_MEMORY_SCOPE_AGENT);
            return nstep;
        }
        __builtin_amdgcn_s_sleep(4);
    }
    return horizon;
}

__device__ __forceinline__ void pack_unit(const Params& p, int u, unsigned char* dst, LAS unsigned char* lds, int wid, int lane) {
    const int j = u & 3;
    if (u < XU) {
        const int pm = u >> 2;
        u32x4 v[4];
#pragma unroll
        for (int i = 0; i < 4; ++i) { const int pc = wid * 4 + i;
            const int row = 256 * pm + 16 * (pc >> 1) + (lane & 15), col = 128 * j + 64 * (pc & 1) + 16 * (lane >> 4);
            v[i] = *(const u32x4*)(p.x + (size_t)row * KSH + col); }
#pragma unroll
        for (int i = 0; i < 4; ++i) st16_sys(dst + (wid * 4 + i) * 1024 + lane * 16, v[i]);
    } else {
        const int pn = (u - XU) >> 2;
        const unsigned char* src = p.w + (size_t)(128 * j + 16 * wid) * N_ + 256 * pn + 4 * lane;
        unsigned r[16];
#pragma unroll
        for (int i = 0; i < 16; ++i) r[i] = *(const unsigned*)(src + (size_t)i * N_);
        u32x4 o[4];
#pragma unroll
        for (int q = 0; q < 4; ++q) {
            const unsigned a = r[4 * q], b = r[4 * q + 1], c = r[4 * q + 2], d = r[4 * q + 3];
            const unsigned ab_lo = __builtin_amdgcn_perm(b, a, 0x05010400u), ab_hi = __builtin_amdgcn_perm(b, a, 0x07030602u);
            const unsigned cd_lo = __builtin_amdgcn_perm(d, c, 0x05010400u), cd_hi = __builtin_amdgcn_perm(d, c, 0x07030602u);
            o[0][q] = __builtin_amdgcn_perm(cd_lo, ab_lo, 0x05040100u);
            o[1][q] = __builtin_amdgcn_perm(cd_lo, ab_lo, 0x07060302u);
            o[2][q] = __builtin_amdgcn_perm(cd_hi, ab_hi, 0x05040100u);
            o[3][q] = __builtin_amdgcn_perm(cd_hi, ab_hi, 0x07060302u);
        }
        const unsigned lo = (unsigned)(lane >> 2) * 2048u + (unsigned)(wid >> 2) * 1024u + (unsigned)((wid & 3) * 16 + (lane & 3) * 4) * 16u;
#pragma unroll
        for (int c = 0; c < 4; ++c) *(LAS u32x4*)(lds + lo + c * 16) = o[c];
        WAIT_L(0); BAR; asm volatile("" ::: "memory");
        u32x4 v[4];
#pragma unroll
        for (int i = 0; i < 4; ++i) v[i] = *(const LAS u32x4*)(lds + (wid * 4 + i) * 1024 + lane * 16);
#pragma unroll
        for (int i = 0; i < 4; ++i) st16_sys(dst + (wid * 4 + i) * 1024 + lane * 16, v[i]);
    }
}

__device__ __forceinline__ f32x2 silu2(f32x2 a, float sl, float is) {
    constexpr float LIM = 60.0f * 1.44269504088896340736f;
    const f32x2 t = a * sl;
    f32x2 e;
    e.x = __builtin_amdgcn_exp2f(__builtin_amdgcn_fmed3f(t.x, -LIM, LIM));
    e.y = __builtin_amdgcn_exp2f(__builtin_amdgcn_fmed3f(t.y, -LIM, LIM));
    const f32x2 d = __builtin_elementwise_fma(e, (f32x2){is, is}, (f32x2){is, is});
    f32x2 r; r.x = __builtin_amdgcn_rcpf(d.x); r.y = __builtin_amdgcn_rcpf(d.y);
    return a * r;
}

__global__ __launch_bounds__(NTHR, 2) void fused_kernel(const Params p) {
    extern __shared__ __attribute__((aligned(16))) unsigned char shm[];
    LAS unsigned char* lds = (LAS unsigned char*)shm;
    const int tid = threadIdx.x, wid = __builtin_amdgcn_readfirstlane(tid >> 6), lane = tid & 63;
    const int bid = blockIdx.x, xcd = bid & 7, slot = bid >> 3;
    const int rank = p.rank;
    unsigned char* ws = p.ws[rank];
    unsigned* tmo = (unsigned*)(ws + WS_TMO);

    STAMP(0);
    LAS unsigned* lw = (LAS unsigned*)(lds + 8 * HTB);
    if (tid == 0) lw[0] = __hip_atomic_fetch_add((unsigned*)(ws + WS_CTR + bid * 64), 1u, __ATOMIC_RELAXED, __HIP_MEMORY_SCOPE_AGENT) + 1u;
    __syncthreads();
    const unsigned call = __builtin_amdgcn_readfirstlane(lw[0]);
    const unsigned stage_off = WS_STAGE + (call & 1u) * PAR_BYTES;
    unsigned char* stage = ws + stage_off;

    if (!(p.mode & 1)) {
        for (int rep = 0; rep < 2; ++rep) {
            int o = bid;
            if (rep == 1) { if (slot < EX || slot >= EX + 8) break; o = NWG + (slot - EX) * 8 + xcd; }
            const int u = unit_of(o, 0);
            pack_unit(p, u, stage + (size_t)rank * SRC_BYTES + (size_t)u * UNIT, lds, wid, lane);
            WAIT_V(0); BAR; asm volatile("" ::: "memory");
            if (tid < NR) {
                if (tid == rank) __hip_atomic_store((unsigned*)(ws + WS_LF) + lf_index(u, rank), call, __ATOMIC_RELAXED, __HIP_MEMORY_SCOPE_AGENT);
                else __hip_atomic_store((unsigned*)(p.ws[tid] + WS_SF + (unsigned)(rank * NU + u) * 64u), call, __ATOMIC_RELAXED, __HIP_MEMORY_SCOPE_SYSTEM);
            }
        }
    }

    STAMP(1);
    if (slot >= GX) {
        if (p.mode & 2) return;
        constexpr int NITEM = NU * (NR - 1), P = NPULL > 0 ? PULLERS : 16;
        const int c = (slot - GX) * 8 + xcd;
        if (c >= P) return;
        const unsigned toff = (unsigned)wid * 4096u + (unsigned)lane * 16u;
        constexpr int NSTEP = (NITEM + P - 1) / P;
        const int nstep = (NITEM - c + P - 1) / P;
        LAS unsigned* tab = (LAS unsigned*)(lds + 8 * HTB + 64);
        static_assert(NSTEP + 4 <= 512 && 64 + (NSTEP + 4) * 4 <= LDS_BYTES - 8 * (int)HTB, "item table");
        if (tid < NSTEP + 4) { const int k = tid < nstep ? tid : nstep - 1, n = c + k * P, o = n / (NR - 1), q = n - o * (NR - 1);
            tab[tid] = (unsigned)(((rank + 1 + q) & (NR - 1)) << 16) | (unsigned)unit_of(o, ROT ? q : 0); }
        __syncthreads();
        int horizon = 0;
#define DESC(k) ((unsigned)__builtin_amdgcn_readfirstlane(tab[k]))
#define PULL_LOAD(k) do { \
            if ((k) >= horizon && (k) < nstep) horizon = pull_horizon(tab, horizon, (k), nstep, ws, call, tmo, lane); \
            const unsigned d_ = DESC(k), s_ = (d_ >> 16) & (NR - 1), u_ = d_ & 0xffffu; \
            const unsigned char* g_ = p.ws[s_] + stage_off + (size_t)s_ * SRC_BYTES + (size_t)u_ * UNIT; \
            _Pragma("unroll") for (int i_ = 0; i_ < 4; ++i_) \
                __builtin_amdgcn_global_load_lds((const unsigned*)(g_ + i_ * 1024 + toff), (LAS unsigned*)(lds + ((k) & 3) * UNIT + wid * 4096 + i_ * 1024), 16, 0, PULL_AUX); } while (0)
#define PULL_PUBLISH(k) do { if (tid == 0) { const unsigned d_ = tab[k]; \
            __hip_atomic_store((unsigned*)(ws + WS_LF) + lf_index((int)(d_ & 0xffffu), (int)(d_ >> 16)), call, __ATOMIC_RELAXED, __HIP_MEMORY_SCOPE_AGENT); } } while (0)
        PULL_LOAD(0); PULL_LOAD(1);
        for (int k = 0; k < nstep; ++k) {
            PULL_LOAD(k + 2);
            if (k == 0) WAIT_V(8); else WAIT_V(12);
            const unsigned d = DESC(k), s = (d >> 16) & (NR - 1), u = d & 0xffffu;
            unsigned char* dst = stage + (size_t)s * SRC_BYTES + (size_t)u * UNIT + toff;
            const unsigned la = (unsigned)(size_t)(lds + (k & 3) * UNIT + toff);
            u32x4 t0, t1, t2, t3;
            asm volatile("ds_read_b128 %0, %4\n\tds_read_b128 %1, %4 offset:1024\n\tds_read_b128 %2, %4 offset:2048\n\tds_read_b128 %3, %4 offset:3072\n\t"
                         "s_waitcnt lgkmcnt(3)\n\tglobal_store_dwordx4 %5, %0, off " PULL_ST "\n\t"
                         "s_waitcnt lgkmcnt(2)\n\tglobal_store_dwordx4 %5, %1, off offset:1024 " PULL_ST "\n\t"
                         "s_waitcnt lgkmcnt(1)\n\tglobal_store_dwordx4 %5, %2, off offset:2048 " PULL_ST "\n\t"
                         "s_waitcnt lgkmcnt(0)\n\tglobal_store_dwordx4 %5, %3, off offset:3072 " PULL_ST "\n\ts_nop 1"
                         : "=&v"(t0), "=&v"(t1), "=&v"(t2), "=&v"(t3) : "v"(la), "v"(dst) : "memory");
            BAR; asm volatile("" ::: "memory");
            if (k >= 2) PULL_PUBLISH(k - 2);
        }
        WAIT_V(0); BAR; asm volatile("" ::: "memory");
        if (nstep >= 2) PULL_PUBLISH(nstep - 2);
        PULL_PUBLISH(nstep - 1);
        STAMP(2);
        return;
    }

    if (p.mode & 4) return;
    const int wr = wid >> 2, wc = wid & 3;
    int kt_l; { const int s = lane & 31; if (s < 4) kt_l = rank * 4 + s; else { const int j = (s - 4) / 7, q = (s - 4) - j * 7; kt_l = ((rank + 1 + q) & 7) * 4 + j; } }
    const int koff = (int)((unsigned)(kt_l >> 2) * SRC_BYTES + (unsigned)(kt_l & 3) * UNIT);
    const unsigned* lf = (const unsigned*)(ws + WS_LF);
    const unsigned voff = (unsigned)tid * 16u;
    const unsigned ldsw = (unsigned)wid * 1024u;
    const int aoff = wr * 8192 + lane * 16, boff = wc * 4096 + lane * 16;
#define SA(b, h) (((b) * 2 + (h)) * HTB)
#define SB(b, h) ((4 + (b) * 2 + (h)) * HTB)
#define STAGE(bufoff, gbase) do { _Pragma("unroll") for (int _i = 0; _i < 2; ++_i) \
        __builtin_amdgcn_global_load_lds((const unsigned*)((gbase) + _i * 8192 + voff), (LAS unsigned*)(lds + (bufoff) + ldsw + _i * 8192), 16, 0, 0); } while (0)
#define LDFRAG(off) __builtin_shufflevector(*(const LAS i32x4*)(lds + (off)), *(const LAS i32x4*)(lds + (off) + 1024), 0, 1, 2, 3, 4, 5, 6, 7)
#define LDA(dst, b, h) do { _Pragma("unroll") for (int m = 0; m < 4; ++m) dst[m] = LDFRAG(SA(b, h) + aoff + m * 2048); } while (0)
#define LDB(dst, b, h) do { _Pragma("unroll") for (int n = 0; n < 2; ++n) dst[n] = LDFRAG(SB(b, h) + boff + n * 2048); } while (0)
#define MMA(ai, bj, At, Bt) do { __builtin_amdgcn_s_setprio(1); _Pragma("unroll") for (int m = 0; m < 4; ++m) _Pragma("unroll") for (int n = 0; n < 2; ++n) \
        acc[ai][bj][m][n] = __builtin_amdgcn_mfma_scale_f32_16x16x128_f8f6f4(Bt[n], At[m], acc[ai][bj][m][n], 1, 1, 0, 0, 0, 0); __builtin_amdgcn_s_setprio(0); } while (0)
#define KOFF(s) ((unsigned)__builtin_amdgcn_readlane(koff, (s)))
#define FLAGPTR(pm_, pn_) (lf + ((lane < 32) ? (pm_) : 16 + (pn_)) * 32 + kt_l)
#define MASK_OF(f) ({ const unsigned long long b_ = __ballot((f) >= call); (unsigned)b_ & (unsigned)(b_ >> 32); })
#define NEED(pm_, pn_, s) do { if (!((mask >> (s)) & 1u)) { const int k_ = __builtin_amdgcn_readlane(kt_l, (s)); \
        wait_flags<false>(lf + ((lane & 1) ? (pm_) : 16 + (pn_)) * 32 + k_, call, tmo); } } while (0)

    int pm, pn, npm = 0, npn = 0, ti = 0;
    if (!tile_of(xcd, slot, 0, pm, pn)) return;
    f32x4 acc[2][2][4][2];
#pragma unroll
    for (int a = 0; a < 2; ++a)
#pragma unroll
        for (int b = 0; b < 2; ++b)
#pragma unroll
            for (int m = 0; m < 4; ++m)
#pragma unroll
                for (int n = 0; n < 2; ++n) acc[a][b][m][n] = (f32x4){0.f, 0.f, 0.f, 0.f};
    i32x8 At[4], B0[2], B1[2];
    const unsigned char* cA = stage + (size_t)pm * 4 * UNIT;
    const unsigned char* cB = stage + (size_t)(XU + pn * 4) * UNIT;
    unsigned mask = 0, fv = 0, tv = 0;
    const float scale = __builtin_bit_cast(float, __builtin_amdgcn_readfirstlane(__builtin_bit_cast(int, p.sx[0] * p.sw[0])));
    const float sl = -scale * 1.44269504088896340736f, inv_s = 1.0f / scale;
    NEED(pm, pn, 0); NEED(pm, pn, 1);
    { const unsigned k0 = KOFF(0), k1 = KOFF(1);
      ld4_dev_async(fv, FLAGPTR(pm, pn));
      STAGE(SB(0, 0), cB + k0); STAGE(SA(0, 0), cA + k0); STAGE(SB(0, 1), cB + k0 + HTB); STAGE(SA(0, 1), cA + k0 + HTB);
      if (wr == 1) BAR;
      WAIT_V(4); BAR;
      asm volatile("" : "+v"(fv)); mask = MASK_OF(fv);
      STAGE(SB(1, 0), cB + k1); STAGE(SA(1, 0), cA + k1); STAGE(SB(1, 1), cB + k1 + HTB);
      WAIT_V(6); BAR; }
    for (;;) {
        const bool has_next = tile_of(xcd, slot, ti + 1, npm, npn);
        if (!has_next) { npm = pm; npn = pn; }
        const unsigned char* nA = stage + (size_t)npm * 4 * UNIT;
        const unsigned char* nB = stage + (size_t)(XU + npn * 4) * UNIT;
        for (int t = 0; t < 32; t += 2) {
            const bool last = (t == 30);
            const int s2 = (t + 2) & 31, s3 = (t + 3) & 31;
            const unsigned k1 = KOFF(t + 1), k2 = KOFF(s2), k3 = KOFF(s3);
            const unsigned char* a1 = cA + k1;
            const unsigned char* a2 = (last ? nA : cA) + k2; const unsigned char* b2 = (last ? nB : cB) + k2;
            const unsigned char* a3 = (last ? nA : cA) + k3; const unsigned char* b3 = (last ? nB : cB) + k3;
            const int qm = last ? npm : pm, qn = last ? npn : pn;
            const bool r1 = (mask != 0xffffffffu);
            if (r1) ld4_dev_async(fv, FLAGPTR(qm, qn));
            LDB(B0, 0, 0); SCHED; LDA(At, 0, 0); STAGE(SA(1, 1), a1 + HTB);
            WAIT_L(8); BAR; WAIT_L(0); MMA(0, 0, At, B0); BAR; SCHED;
            NEED(qm, qn, s2);
            LDB(B1, 0, 1); STAGE(SB(0, 0), b2);
            BAR; WAIT_L(0); MMA(0, 1, At, B1); BAR;
            LDA(At, 0, 1); STAGE(SA(0, 0), a2);
            BAR; WAIT_L(0); MMA(1, 0, At, B0); BAR; SCHED;
            STAGE(SB(0, 1), b2 + HTB);
            WAIT_V(6);
            if (r1) { asm volatile("" : "+v"(fv)); mask = MASK_OF(fv); }
            BAR; MMA(1, 1, At, B1); BAR;
            const bool sw = (t == 28);
            const bool r5 = sw || (mask != 0xffffffffu);
            if (r5) ld4_dev_async(fv, FLAGPTR(sw ? npm : qm, sw ? npn : qn));
            if (last) ld4_dev_async(tv, tmo);
            LDB(B0, 1, 0); SCHED; LDA(At, 1, 0); STAGE(SA(0, 1), a2 + HTB);
            WAIT_L(8); BAR; WAIT_L(0); MMA(0, 0, At, B0); BAR; SCHED;
            NEED(qm, qn, s3);
            LDB(B1, 1, 1); STAGE(SB(1, 0), b3);
            BAR; WAIT_L(0); MMA(0, 1, At, B1); BAR;
            LDA(At, 1, 1); STAGE(SA(1, 0), a3);
            BAR; WAIT_L(0); MMA(1, 0, At, B0); BAR; SCHED;
            STAGE(SB(1, 1), b3 + HTB);
            WAIT_V(6);
            asm volatile("" : "+v"(tv));
            if (r5) { asm volatile("" : "+v"(fv)); mask = MASK_OF(fv); }
            BAR; MMA(1, 1, At, B1); BAR;
        }
        STAMP(2 + 2 * ti);
        {
            const float is = ((unsigned)__builtin_amdgcn_readfirstlane(tv) == call) ? __builtin_nanf("") : inv_s;
            float* o0 = p.out + (size_t)(pm * 256 + wr * 64 + (lane & 15)) * N_ + pn * 256 + wc * 32 + 4 * (lane >> 4);
#pragma unroll
            for (int ai = 0; ai < 2; ++ai)
#pragma unroll
                for (int m = 0; m < 4; ++m) { float* rowp = o0 + (size_t)(ai * 128 + m * 16) * N_;
#pragma unroll
                    for (int bj = 0; bj < 2; ++bj)
#pragma unroll
                        for (int n = 0; n < 2; ++n) { const f32x4 a = acc[ai][bj][m][n];
                            const f32x2 lo = silu2((f32x2){a[0], a[1]}, sl, is), hi = silu2((f32x2){a[2], a[3]}, sl, is);
                            const f32x4 v = {lo.x, lo.y, hi.x, hi.y};
                            asm volatile("global_store_dwordx4 %0, %1, off offset:%2 " OUT_ST "\n\ts_nop 1" :: "v"(rowp), "v"(v), "i"((bj * 128 + n * 16) * 4) : "memory"); } }
        }
        STAMP(3 + 2 * ti);
        if (!has_next) break;
#pragma unroll
        for (int a = 0; a < 2; ++a)
#pragma unroll
            for (int b = 0; b < 2; ++b)
#pragma unroll
                for (int m = 0; m < 4; ++m)
#pragma unroll
                    for (int n = 0; n < 2; ++n) acc[a][b][m][n] = (f32x4){0.f, 0.f, 0.f, 0.f};
        pm = npm; pn = npn; cA = nA; cB = nB; ++ti;
    }
    WAIT_V(0);
    STAMP(12);
    if (wr == 0) BAR;
}

}

extern "C" void kernel_launch(int rank, int nranks, ncclComm_t comm,
                              hipStream_t stream,
                              void* const* d_in, const int* in_sizes, int n_in,
                              void* d_out, int out_size,
                              const int* in_dtypes, int out_dtype,
                              void* const* d_out_all,
                              void* const* d_ws_all, size_t ws_bytes) {
    (void)comm; (void)in_dtypes; (void)out_dtype; (void)d_out_all;
    if (nranks != NR || (unsigned)rank >= (unsigned)NR || n_in != 4 || in_sizes[0] != M_ * KSH || in_sizes[1] != KSH * N_ ||
        out_size != M_ * N_ || ws_bytes < WS_NEED) {
        fprintf(stderr, "kernel_launch: unexpected problem shape\n");
        return;
    }
    HIP_CHECK(hipFuncSetAttribute((const void*)fused_kernel, hipFuncAttributeMaxDynamicSharedMemorySize, LDS_BYTES));
    Params p;
    p.x = (const unsigned char*)d_in[0]; p.w = (const unsigned char*)d_in[1];
    p.sx = (const float*)d_in[2]; p.sw = (const float*)d_in[3];
    p.out = (float*)d_out;
    for (int i = 0; i < NR; ++i) p.ws[i] = (unsigned char*)d_ws_all[i];
    p.rank = rank;
    p.mode = 0;
    hipLaunchKernelGGL(fused_kernel, dim3(NWG), dim3(NTHR), LDS_BYTES, stream, p);
    HIP_CHECK(hipGetLastError());
}
```

```cpp
#include <hip/hip_runtime.h>
#include <rccl/rccl.h>
#include <cstdio>

#define HIP_CHECK(expr) do { hipError_t _e = (expr); if (_e != hipSuccess) { \
    fprintf(stderr, "HIP error %s at %s:%d\n", hipGetErrorString(_e), __FILE__, __LINE__); \
    return; } } while (0)

namespace {

#define LAS __attribute__((address_space(3)))
typedef int i32x4 __attribute__((ext_vector_type(4)));
typedef int i32x8 __attribute__((ext_vector_type(8)));
typedef unsigned u32x4 __attribute__((ext_vector_type(4)));
typedef float f32x4 __attribute__((ext_vector_type(4)));
typedef float f32x2 __attribute__((ext_vector_type(2)));

constexpr int M_ = 4096, N_ = 16384, NR = 8, KSH = 512;
constexpr int NWG = 256, NTHR = 512;
constexpr unsigned UNIT = 32768;
constexpr unsigned HTB = 16384;
constexpr int NPANEL = 80, NU = 4 * NPANEL;
constexpr unsigned SRC_BYTES = NU * UNIT;
constexpr unsigned PAR_BYTES = NR * SRC_BYTES;
constexpr unsigned WS_CTR = 0;
constexpr unsigned WS_TMO = 16384;
constexpr unsigned WS_TK = 20480;
constexpr unsigned WS_LF = 32768;
constexpr unsigned WS_SF = 65536;
constexpr unsigned WS_DBG = 512 * 1024;
constexpr unsigned WS_STAGE = 1u << 20;
constexpr size_t WS_NEED = (size_t)WS_STAGE + 2ull * PAR_BYTES;
constexpr unsigned long long DEADLINE_TICKS = 40000000ull;

#define WPP 3
#define PULL_LA 2
#define TAILM 0
#define SEG2 1
#define MORD 1
#define OUT_ST "nt"
constexpr int NPULL = 7 * WPP;
constexpr int MAXT = 5, NSEG = 5;
constexpr int LDS_CTL = 8 * HTB, LDS_BYTES = LDS_CTL + 1024;
constexpr int ROLE_GEMM = 127, ROLE_PACK2 = 32;

struct Params {
    const unsigned char* x; const unsigned char* w; const float* sx; const float* sw;
    float* out;
    unsigned char* ws[NR];
    int rank; int mode;
    int seg[NSEG + 1];
    unsigned char panel[NPANEL];
    unsigned char pidx[NPANEL];
    unsigned rec[NWG][3];
};
static_assert(sizeof(Params) <= 4096, "kernel arguments");

__host__ __device__ __forceinline__ int unit_of(const Params& p, int o, int q) {
    int s0 = 0, n = 1;
#pragma unroll
    for (int s = 0; s < NSEG; ++s) if (o >= 4 * p.seg[s] && p.seg[s + 1] > p.seg[s]) { s0 = p.seg[s]; n = p.seg[s + 1] - s0; }
    o -= 4 * s0;
    const int j = (o >= n) + (o >= 2 * n) + (o >= 3 * n);
    int i = o - j * n + (q * n) / 7; if (i >= n) i -= n;
    return (int)p.panel[s0 + i] * 4 + j;
}

#define WAIT_V(n) asm volatile("s_waitcnt vmcnt(" #n ")" ::: "memory")
#define WAIT_L(n) asm volatile("s_waitcnt lgkmcnt(" #n ")" ::: "memory")
#define BAR __builtin_amdgcn_s_barrier()
#define FENCE asm volatile("" ::: "memory")
#define SCHED __builtin_amdgcn_sched_barrier(0)
#define STAMP(i) do { } while (0)

__device__ __forceinline__ void st16_sys(void* p, u32x4 v) { asm volatile("global_store_dwordx4 %0, %1, off sc0 sc1\n\ts_nop 1" :: "v"(p), "v"(v) : "memory"); }
__device__ __forceinline__ void ld4_dev_async(unsigned& v, const void* base, unsigned off) { asm volatile("global_load_dword %0, %1, %2 sc1" : "+v"(v) : "v"(off), "s"(base) : "memory"); }
__device__ __forceinline__ unsigned ld4_sys_now(const void* p) { unsigned v; asm volatile("global_load_dword %0, %1, off sc0 sc1\n\ts_waitcnt vmcnt(0)" : "=&v"(v) : "v"(p) : "memory"); return v; }
__device__ __forceinline__ unsigned ld4_dev_now(const void* p) { unsigned v; asm volatile("global_load_dword %0, %1, off sc1\n\ts_waitcnt vmcnt(0)" : "=&v"(v) : "v"(p) : "memory"); return v; }

__device__ __forceinline__ bool give_up(unsigned long long t0, unsigned call, unsigned* tmo) {
    if ((unsigned)__builtin_amdgcn_readfirstlane(ld4_dev_now(tmo)) == call) return true;
    if (__builtin_amdgcn_s_memrealtime() - t0 > DEADLINE_TICKS) {
        __hip_atomic_store(tmo, call, __ATOMIC_RELAXED, __HIP_MEMORY_SCOPE_AGENT);
        WAIT_V(0);
        return true;
    }
    return false;
}
__device__ __noinline__ void wait_flags(const unsigned* flag, unsigned call, unsigned* tmo) {
    const unsigned long long t0 = __builtin_amdgcn_s_memrealtime();
    for (unsigned it = 1;; ++it) {
        if (__all(ld4_dev_now(flag) >= call)) break;
        if ((it & 7u) == 0u && give_up(t0, call, tmo)) break;
        __builtin_amdgcn_s_sleep(12);
    }
}

__device__ __forceinline__ unsigned lf_index(int u, int src) {
    return (unsigned)(u >> 2) * 32u + (unsigned)(src * 4 + (u & 3));
}

__device__ __forceinline__ void pack_load(const Params& p, int u, u32x4 (&o)[4], int wid, int lane) {
    const int j = u & 3, pa = u >> 2;
    if (pa < 16) {
#pragma unroll
        for (int i = 0; i < 4; ++i) { const int pc = wid * 4 + i;
            const int row = 256 * pa + 16 * (pc >> 1) + (lane & 15), col = 128 * j + 64 * (pc & 1) + 16 * (lane >> 4);
            o[i] = *(const u32x4*)(p.x + (size_t)row * KSH + col); }
    } else {
        const unsigned char* src = p.w + (size_t)(128 * j + 16 * wid) * N_ + 256 * (pa - 16) + 4 * lane;
        unsigned r[16];
#pragma unroll
        for (int i = 0; i < 16; ++i) r[i] = *(const unsigned*)(src + (size_t)i * N_);
#pragma unroll
        for (int q = 0; q < 4; ++q) {
            const unsigned a = r[4 * q], b = r[4 * q + 1], c = r[4 * q + 2], d = r[4 * q + 3];
            const unsigned ab_lo = __builtin_amdgcn_perm(b, a, 0x05010400u), ab_hi = __builtin_amdgcn_perm(b, a, 0x07030602u);
            const unsigned cd_lo = __builtin_amdgcn_perm(d, c, 0x05010400u), cd_hi = __builtin_amdgcn_perm(d, c, 0x07030602u);
            o[0][q] = __builtin_amdgcn_perm(cd_lo, ab_lo, 0x05040100u);
            o[1][q] = __builtin_amdgcn_perm(cd_lo, ab_lo, 0x07060302u);
            o[2][q] = __builtin_amdgcn_perm(cd_hi, ab_hi, 0x05040100u);
            o[3][q] = __builtin_amdgcn_perm(cd_hi, ab_hi, 0x07060302u);
        }
    }
}
__device__ __forceinline__ void pack_store(int u, unsigned char* dst, const u32x4 (&o)[4], int wid, int lane) {
    const bool isx = (u >> 2) < 16;
    const unsigned base = isx ? (unsigned)wid * 4096u + (unsigned)lane * 16u
                              : (unsigned)(lane >> 4) * 4096u + (unsigned)(wid >> 2) * 1024u + (unsigned)((wid & 3) * 16 + (lane & 15)) * 16u;
#pragma unroll
    for (int c = 0; c < 4; ++c) st16_sys(dst + base + (isx ? c * 1024u : (c >> 1) * HTB + (c & 1) * 2048u), o[c]);
}

__device__ __forceinline__ float exp_clip(float t) {
    constexpr float LIM = 60.0f * 1.44269504088896340736f;
    return __builtin_amdgcn_exp2f(__builtin_amdgcn_fmed3f(t, -LIM, LIM));
}
__device__ __forceinline__ f32x4 silu4(float a0, float a1, float a2, float a3, float t0, float t1, float t2, float t3, float is) {
    const f32x2 e01 = {exp_clip(t0), exp_clip(t1)}, e23 = {exp_clip(t2), exp_clip(t3)};
    const f32x2 d01 = __builtin_elementwise_fma(e01, (f32x2){is, is}, (f32x2){is, is}), d23 = __builtin_elementwise_fma(e23, (f32x2){is, is}, (f32x2){is, is});
    float r0, r1, r2, r3;
    asm("v_rcp_f32 %0, %4\n\tv_rcp_f32 %1, %5\n\tv_rcp_f32 %2, %6\n\tv_rcp_f32 %3, %7\n\t"
        "v_mul_f32 %0, %8, %0\n\tv_mul_f32 %1, %9, %1\n\tv_mul_f32 %2, %10, %2\n\tv_mul_f32 %3, %11, %3"
        : "=&v"(r0), "=&v"(r1), "=&v"(r2), "=&v"(r3) : "v"(d01.x), "v"(d01.y), "v"(d23.x), "v"(d23.y), "v"(a0), "v"(a1), "v"(a2), "v"(a3));
    return (f32x4){r0, r1, r2, r3};
}

template <bool B> struct BoolC { static constexpr bool value = B; };

__global__ __launch_bounds__(NTHR, 2) void fused_kernel(const Params p) {
    extern __shared__ __attribute__((aligned(16))) unsigned char shm[];
    LAS unsigned char* lds = (LAS unsigned char*)shm;
    LAS unsigned* ctl = (LAS unsigned*)(lds + LDS_CTL);
    const int tid = threadIdx.x, wid = __builtin_amdgcn_readfirstlane(tid >> 6), lane = tid & 63;
    const int bid = blockIdx.x;
    const int rank = p.rank;
    unsigned char* ws = p.ws[rank];
    unsigned* tmo = (unsigned*)(ws + WS_TMO);
    STAMP(0);

    unsigned ticket = 0;
    if (tid == 0) ticket = __hip_atomic_fetch_add((unsigned*)(ws + WS_CTR + bid * 64), 1u, __ATOMIC_RELAXED, __HIP_MEMORY_SCOPE_AGENT);
    const unsigned rec0 = p.rec[bid][0], rec1 = p.rec[bid][1], rec2 = p.rec[bid][2];
    const int role = (int)(rec2 >> 25);
    int pu = (int)((rec2 >> 16) & 511u);
    u32x4 pk[4];
    pack_load(p, pu, pk, wid, lane);
    if (tid == 0) ctl[0] = ticket + 1u;
    __syncthreads();
    const unsigned call = __builtin_amdgcn_readfirstlane(ctl[0]);
    const unsigned stage_off = WS_STAGE + (call & 1u) * PAR_BYTES;
    unsigned char* stage = ws + stage_off;

    if (!(p.mode & 1)) {
        for (int rep = 0;; ++rep) {
            pack_store(pu, stage + (size_t)rank * SRC_BYTES + (size_t)pu * UNIT, pk, wid, lane);
            WAIT_V(0); BAR; FENCE;
            if (tid < NR) {
                if (tid == rank) __hip_atomic_store((unsigned*)(ws + WS_LF) + lf_index(pu, rank), call, __ATOMIC_RELAXED, __HIP_MEMORY_SCOPE_AGENT);
                else __hip_atomic_store((unsigned*)(p.ws[tid] + WS_SF + (unsigned)(rank * NU + pu) * 64u), call, __ATOMIC_RELAXED, __HIP_MEMORY_SCOPE_SYSTEM);
            }
            if (rep == 1 || role < ROLE_PACK2 || role >= ROLE_PACK2 + (NU - NWG)) break;
            pu = __builtin_amdgcn_readfirstlane(unit_of(p, NWG + role - ROLE_PACK2, 0));
            pack_load(p, pu, pk, wid, lane);
        }
    }
    STAMP(1);

    if (role < NPULL) {
        if (p.mode & 2) return;
        const int q = role % 7, wq = role / 7, src = (rank + 1 + q) & (NR - 1);
        constexpr int C_PLAN = 2, C_ISSUED = 4, C_LANDED = 6, C_FREED = 8, C_DESC = 16, C_CNT = 32, C_TAB = 48;
        LAS unsigned short* utab = (LAS unsigned short*)(ctl + C_TAB);
        if (tid < NU) utab[tid] = (unsigned short)unit_of(p, tid, q);
        if (tid >= 2 && tid < C_TAB) ctl[tid] = 0u;
        __syncthreads();
        const unsigned cbase = (unsigned)(size_t)ctl;
#define CGET(w) ({ int v_; asm volatile("ds_read_b32 %0, %1\n\ts_waitcnt lgkmcnt(0)" : "=&v"(v_) : "v"(cbase + (unsigned)(w) * 4u) : "memory"); __builtin_amdgcn_readfirstlane(v_); })
#define CPUT(w, v) asm volatile("ds_write_b32 %0, %1" :: "v"(cbase + (unsigned)(w) * 4u), "v"((int)(v)) : "memory")
        const unsigned long long tstart = __builtin_amdgcn_s_memrealtime();
#define SPIN_GUARD() do { __builtin_amdgcn_s_sleep(1); if (__builtin_amdgcn_s_memrealtime() - tstart > 2 * DEADLINE_TICKS) { \
            __hip_atomic_store(tmo, call, __ATOMIC_RELAXED, __HIP_MEMORY_SCOPE_AGENT); return; } } while (0)
        if (wid < 2) {
            const unsigned char* rbase = p.ws[src] + stage_off + (size_t)src * SRC_BYTES + (unsigned)wid * HTB;
            const unsigned loff = (unsigned)lane * 16u;
            int landed = 0;
            for (int k = 0;; ++k) {
                for (;;) {
                    const int f0 = CGET(C_FREED + 2 * wid), f1 = CGET(C_FREED + 2 * wid + 1);
                    if (CGET(C_PLAN) > k && f0 + 4 > k && f1 + 4 > k) break;
                    if (landed < k) { WAIT_V(0); landed = k; CPUT(C_LANDED + wid, k); } else SPIN_GUARD();
                }
                const int u = CGET(C_DESC + (k & 15));
                if (u < 0) { WAIT_V(0); CPUT(C_LANDED + wid, k); break; }
                const unsigned char* g = rbase + (size_t)u * UNIT;
#pragma unroll
                for (int i = 0; i < 16; ++i)
                    __builtin_amdgcn_global_load_lds((const unsigned*)(g + i * 1024 + loff), (LAS unsigned*)(lds + (k & 3) * UNIT + wid * HTB + i * 1024), 16, 0, 17  );
                CPUT(C_ISSUED + wid, k + 1);
                if (PULL_LA == 2) WAIT_V(16); else WAIT_V(32);
                if (landed < k + 2 - PULL_LA) { landed = k + 2 - PULL_LA; CPUT(C_LANDED + wid, landed); }
            }
            STAMP(2);
        } else if (wid == 2) {
            unsigned* tk = (unsigned*)(ws + WS_TK + q * 128);
            const unsigned tk_base = (call - 1u) * (unsigned)(NU - 2 * WPP);
            int horizon = 0;
            for (int k = 0;; ++k) {
                while (k > CGET(C_ISSUED) + 1 || k > CGET(C_ISSUED + 1) + 1) SPIN_GUARD();
                int pos;
                if (k < 3) pos = k * WPP + wq;
                else { unsigned t = 0; if (lane == 0) t = __hip_atomic_fetch_add(tk, 1u, __ATOMIC_RELAXED, __HIP_MEMORY_SCOPE_AGENT);
                    const unsigned r = (unsigned)__builtin_amdgcn_readfirstlane(t) - tk_base;
                    pos = r < (unsigned)(NU - 3 * WPP) ? 3 * WPP + (int)r : -1; }
                if (pos >= horizon) {
                    const unsigned long long t0 = __builtin_amdgcn_s_memrealtime();
                    horizon = pos;
                    for (;;) {
                        const int o = horizon + lane < NU ? horizon + lane : NU - 1;
                        const unsigned long long nr = ~__ballot(ld4_sys_now(ws + WS_SF + (unsigned)(src * NU + (int)utab[o]) * 64u) >= call);
                        horizon += nr ? __builtin_ctzll(nr) : 64;
                        if (horizon > pos) break;
                        if (give_up(t0, call, tmo)) { horizon = NU; break; }
                        __builtin_amdgcn_s_sleep(2);
                    }
                }
                CPUT(C_DESC + (k & 15), pos >= 0 ? (int)utab[pos] : -1);
                CPUT(C_PLAN, k + 1);
                if (pos < 0) break;
            }
        } else if (wid >= 4) {
            const int i4 = wid - 4;
            const unsigned toff = (unsigned)i4 * 8192u + (unsigned)lane * 16u;
            unsigned char* lbase = stage + (size_t)src * SRC_BYTES + toff;
            int pend = -1, pend_u = 0;
#define ARRIVE() do { if (lane == 0) { unsigned old_; asm volatile("ds_add_rtn_u32 %0, %1, %2\n\ts_waitcnt lgkmcnt(0)" : "=&v"(old_) : "v"(cbase + (unsigned)(C_CNT + (pend & 15)) * 4u), "v"(1u) : "memory"); \
                if ((old_ & 3u) == 3u) { __hip_atomic_store((unsigned*)(ws + WS_LF) + lf_index(pend_u, src), call, __ATOMIC_RELAXED, __HIP_MEMORY_SCOPE_AGENT); PUBSTAMP(); } } } while (0)
#define PUBSTAMP() do { } while (0)
            for (int k = 0;; ++k) {
                bool end = false;
                while (CGET(C_LANDED + (i4 >> 1)) <= k) {
                    if (pend >= 0) { WAIT_V(0); ARRIVE(); pend = -1; }
                    else if (CGET(C_PLAN) > k && CGET(C_DESC + (k & 15)) < 0) { end = true; break; }
                    else SPIN_GUARD();
                }
                if (end) break;
                const int u = CGET(C_DESC + (k & 15));
                unsigned char* dst = lbase + (size_t)u * UNIT;
                const unsigned la = (unsigned)(size_t)(lds + (k & 3) * UNIT + toff);
#pragma unroll
                for (int h = 0; h < 2; ++h) { u32x4 t0, t1, t2, t3;
                    asm volatile("ds_read_b128 %0, %4\n\tds_read_b128 %1, %4 offset:1024\n\tds_read_b128 %2, %4 offset:2048\n\tds_read_b128 %3, %4 offset:3072\n\t"
                                 "s_waitcnt lgkmcnt(3)\n\tglobal_store_dwordx4 %5, %0, off sc1\n\t"
                                 "s_waitcnt lgkmcnt(2)\n\tglobal_store_dwordx4 %5, %1, off offset:1024 sc1\n\t"
                                 "s_waitcnt lgkmcnt(1)\n\tglobal_store_dwordx4 %5, %2, off offset:2048 sc1\n\t"
                                 "s_waitcnt lgkmcnt(0)\n\tglobal_store_dwordx4 %5, %3, off offset:3072 sc1\n\ts_nop 1"
                                 : "=&v"(t0), "=&v"(t1), "=&v"(t2), "=&v"(t3) : "v"(la + h * 4096), "v"(dst + h * 4096) : "memory"); }
                CPUT(C_FREED + i4, k + 1);
                if (pend >= 0) { WAIT_V(8); ARRIVE(); }
                pend = k; pend_u = u;
            }
        }
        return;
    }

    if (p.mode & 4) return;
    const int wr = wid >> 2, wc = wid & 3;
    int tdesc = -1;
    { const unsigned rw = lane < 2 ? rec0 : (lane < 4 ? rec1 : rec2), t = lane < MAXT ? (rw >> (16 * (lane & 1))) & 0xffffu : 0xffffu;
      if (t != 0xffffu) {
          const int ia = p.pidx[t >> 6], ib = p.pidx[16 + (t & 63u)], hi = ia > ib ? ia : ib, lo = ia > ib ? ib : ia;
          int s0 = 0, n = 1;
#pragma unroll
          for (int s = 0; s < NSEG; ++s) if (hi >= p.seg[s] && p.seg[s + 1] > p.seg[s]) { s0 = p.seg[s]; n = p.seg[s + 1] - s0; }
          int key[7];
#pragma unroll
          for (int q = 0; q < 7; ++q) { const int r = (q * n) / 7; int kh = hi - s0 - r; if (kh < 0) kh += n; int kl = lo - s0 - r; if (kl < 0) kl += n;
              key[q] = ((lo >= s0 && kl > kh) ? kl : kh) * 8 + q; }
          unsigned w = 0;
#pragma unroll
          for (int q = 0; q < 7; ++q) { int rk = 0;
#pragma unroll
              for (int r = 0; r < 7; ++r) rk += key[r] < key[q];
              w |= (unsigned)q << (3 * rk); }
          tdesc = (int)(((t >> 6) << 27) | ((t & 63u) << 21) | w);
      } }
    const int ks = lane & 31, kj = ks < 4 ? ks : (ks - 4) / 7, ksh = ks < 4 ? 0 : 3 * ((ks - 4) - kj * 7);
#define KT_OF(d) (ks < 4 ? rank * 4 + ks : ((rank + 1 + (((d) >> ksh) & 7)) & 7) * 4 + kj)
#define PM_OF(d) (((d) >> 27) & 15)
#define PN_OF(d) (((d) >> 21) & 63)
#define FOFF_OF(d, kt) ((unsigned)(((lane < 32) ? PM_OF(d) : 16 + PN_OF(d)) * 32 + (kt)) * 4u)
    const unsigned* lf = (const unsigned*)(ws + WS_LF);
    const unsigned voff = (unsigned)tid * 16u;
    const unsigned ldsw = (unsigned)wid * 1024u;
    const LAS unsigned char* la = lds + wr * 8192 + lane * 16;
    const LAS unsigned char* lb = lds + 4 * HTB + wc * 4096 + lane * 16;
#define SA(b, h) (((b) * 2 + (h)) * HTB)
#define SB(b, h) ((4 + (b) * 2 + (h)) * HTB)
#define ABL_NST 2
#define STAGE(bufoff, gbase) do { _Pragma("unroll") for (int _i = 0; _i < ABL_NST; ++_i) \
        __builtin_amdgcn_global_load_lds((const unsigned*)((gbase) + _i * 8192 + voff), (LAS unsigned*)(lds + (bufoff) + ldsw + _i * 8192), 16, 0, 0); } while (0)
#define LDFRAG(ptr) __builtin_shufflevector(*(const LAS i32x4*)(ptr), *(const LAS i32x4*)((ptr) + 1024), 0, 1, 2, 3, 4, 5, 6, 7)
#define ABL_NLA 4
#define LDA(dst, b, h) do { _Pragma("unroll") for (int m = 0; m < ABL_NLA; ++m) dst[m] = LDFRAG(la + SA(b, h) + m * 2048); } while (0)
#define LDB(dst, b, h) do { _Pragma("unroll") for (int n = 0; n < 2; ++n) dst[n] = LDFRAG(lb + SA(b, h) + n * 2048); } while (0)
#define MFMA1(ai, bj, At, Bt, Z, i) do { constexpr int m = (i) >> 1, n = (i) & 1; \
        if constexpr (Z) asm volatile("v_mfma_f32_16x16x128_f8f6f4 %0, %1, %2, 0 cbsz:1 blgp:1" : "+v"(acc[ai][bj][m][n]) : "v"(At[m]), "v"(Bt[n])); \
        else acc[ai][bj][m][n] = __builtin_amdgcn_mfma_scale_f32_16x16x128_f8f6f4(At[m], Bt[n], acc[ai][bj][m][n], 1, 1, 0, 0, 0, 0); } while (0)
#define MMA(ai, bj, At, Bt, Z, B) do { __builtin_amdgcn_s_setprio(1); \
        MFMA1(ai, bj, At, Bt, Z, 0); MFMA1(ai, bj, At, Bt, Z, 1); MFMA1(ai, bj, At, Bt, Z, 2); MFMA1(ai, bj, At, Bt, Z, 3); MFMA1(ai, bj, At, Bt, Z, 4); \
        if (TAILM == 3) { __builtin_amdgcn_s_setprio(2); SCHED; B; SCHED; } \
        MFMA1(ai, bj, At, Bt, Z, 5); \
        if (TAILM == 2) { __builtin_amdgcn_s_setprio(2); SCHED; B; SCHED; } \
        MFMA1(ai, bj, At, Bt, Z, 6); \
        if (TAILM == 1) { __builtin_amdgcn_s_setprio(2); SCHED; B; SCHED; } \
        MFMA1(ai, bj, At, Bt, Z, 7); \
        __builtin_amdgcn_s_setprio(0); \
        if (TAILM == 0) { B; } \
        SCHED; } while (0)
#define KT(s) __builtin_amdgcn_readlane(kta, (s))
#define KOFF(kt) ((unsigned)((kt) >> 2) * SRC_BYTES + (unsigned)((kt) & 3) * UNIT)
#define MASK_OF(f) ({ const unsigned long long b_ = __ballot((f) >= call); (unsigned)b_ & (unsigned)(b_ >> 32); })
#define NEED(d, s, kt) do { if (!((mask >> (s)) & 1u)) wait_flags(lf + ((lane & 1) ? PM_OF(d) : 16 + PN_OF(d)) * 32 + (kt), call, tmo); } while (0)

    int ti = 0, cur = __builtin_amdgcn_readlane(tdesc, 0);
    if (cur < 0) return;
    f32x4 acc[2][2][4][2] = {};
    i32x8 At[4] = {}, B0[2], B1[2];
    const unsigned char* cA = stage + (size_t)PM_OF(cur) * 4 * UNIT;
    const unsigned char* cB = stage + (size_t)(16 + PN_OF(cur)) * 4 * UNIT;
    int kta = KT_OF(cur);
    unsigned foff = FOFF_OF(cur, kta), mask = 0, fa = 0, fb = 0, tv = 0;
    const float scale = __builtin_bit_cast(float, __builtin_amdgcn_readfirstlane(__builtin_bit_cast(int, p.sx[0] * p.sw[0])));
    const float sl = -scale * 1.44269504088896340736f, inv_s = 1.0f / scale;
    NEED(cur, 0, rank * 4); NEED(cur, 1, rank * 4 + 1);
    { const unsigned k0 = KOFF(rank * 4), k1 = k0 + UNIT;
      ld4_dev_async(fb, lf, foff);
      STAGE(SB(0, 0), cB + k0); STAGE(SA(0, 0), cA + k0); STAGE(SB(0, 1), cB + k0 + HTB); STAGE(SA(0, 1), cA + k0 + HTB);
      STAGE(SB(1, 0), cB + k1); STAGE(SA(1, 0), cA + k1); STAGE(SB(1, 1), cB + k1 + HTB);
      WAIT_V(0); BAR;
      if (wr == 1) BAR; }
    bool rb = true;
    for (;;) {
        int nxt = __builtin_amdgcn_readlane(tdesc, ti + 1);
        const bool has_next = nxt >= 0;
        if (!has_next) nxt = cur;
        const unsigned char* nA = stage + (size_t)PM_OF(nxt) * 4 * UNIT;
        const unsigned char* nB = stage + (size_t)(16 + PN_OF(nxt)) * 4 * UNIT;
        auto trip = [&](auto first_c, const int t) __attribute__((always_inline)) {
            constexpr bool first = decltype(first_c)::value;
#define WAIT_K() do { if constexpr (first) WAIT_V(42); else WAIT_V(10); } while (0)
            const bool last = (t == 30);
            const int s2 = (t + 2) & 31, s3 = (t + 3) & 31;
            const int kt2 = KT(s2), kt3 = KT(s3);
            const unsigned k2 = KOFF(kt2), k3 = KOFF(kt3);
            const unsigned char* a1 = cA + KOFF(KT(t + 1));
            const unsigned char* a2 = (last ? nA : cA) + k2; const unsigned char* b2 = (last ? nB : cB) + k2;
            const unsigned char* a3 = (last ? nA : cA) + k3; const unsigned char* b3 = (last ? nB : cB) + k3;
            const int qd = last ? nxt : cur;
#define WAIT_S() do { if constexpr (first) WAIT_V(40); else WAIT_V(8); } while (0)
#define MMA2(ai, Z, B) do { __builtin_amdgcn_s_setprio(1); \
            MFMA1(ai, 0, At, B0, Z, 0); MFMA1(ai, 0, At, B0, Z, 1); MFMA1(ai, 1, At, B1, Z, 0); MFMA1(ai, 1, At, B1, Z, 1); \
            MFMA1(ai, 1, At, B1, Z, 3); MFMA1(ai, 1, At, B1, Z, 2); MFMA1(ai, 0, At, B0, Z, 3); MFMA1(ai, 0, At, B0, Z, 2); \
            MFMA1(ai, 0, At, B0, Z, 4); MFMA1(ai, 0, At, B0, Z, 5); MFMA1(ai, 1, At, B1, Z, 4); MFMA1(ai, 1, At, B1, Z, 5); \
            MFMA1(ai, 1, At, B1, Z, 7); MFMA1(ai, 1, At, B1, Z, 6); MFMA1(ai, 0, At, B0, Z, 7); MFMA1(ai, 0, At, B0, Z, 6); \
            __builtin_amdgcn_s_setprio(0); B; SCHED; } while (0)
            if (last) ld4_dev_async(tv, tmo, 0u);
            LDB(B0, 0, 0); LDA(At, 0, 0); LDB(B1, 0, 1); STAGE(SA(1, 1), a1 + HTB);
            WAIT_S();
            if (rb) { asm volatile("" : "+v"(fb)); mask = MASK_OF(fb); }
            WAIT_L(0); BAR; MMA2(0, first, BAR);
            NEED(qd, s2, kt2);
            rb = (mask != 0xffffffffu);
            if (rb) ld4_dev_async(fb, lf, foff);
            LDA(At, 0, 1); STAGE(SB(0, 0), b2); STAGE(SA(0, 0), a2); STAGE(SB(0, 1), b2 + HTB);
            WAIT_S(); WAIT_L(0); BAR; MMA2(1, first, BAR);
            LDB(B0, 1, 0); LDA(At, 1, 0); LDB(B1, 1, 1); STAGE(SA(0, 1), a2 + HTB);
            WAIT_V(8);
            if (rb) { asm volatile("" : "+v"(fb)); mask = MASK_OF(fb); }
            WAIT_L(0); BAR; MMA2(0, false, BAR);
            NEED(qd, s3, kt3);
            const bool sw = (t == 28);
            if (sw) foff = FOFF_OF(nxt, KT_OF(nxt));
            rb = sw || (mask != 0xffffffffu);
            if (rb) ld4_dev_async(fb, lf, foff);
            LDA(At, 1, 1); STAGE(SB(1, 0), b3); STAGE(SA(1, 0), a3); STAGE(SB(1, 1), b3 + HTB);
            WAIT_V(8);
            asm volatile("" : "+v"(tv));
            WAIT_L(0); BAR; MMA2(1, false, if (!(last && wr == 1)) BAR);
        };
        trip(BoolC<true>{}, 0);
        for (int t = 2; t < 32; t += 2) trip(BoolC<false>{}, t);
        STAMP(2 + 2 * ti);
        {
            const float is = ((unsigned)__builtin_amdgcn_readfirstlane(tv) == call) ? __builtin_nanf("") : inv_s;
            const float* ob = p.out + (size_t)(PM_OF(cur) * 256 + wr * 64) * N_ + PN_OF(cur) * 256 + wc * 64;
            const unsigned oo = (unsigned)(4 * (lane >> 4)) * (N_ * 4u) + (unsigned)(lane & 15) * 16u;
#pragma unroll
            for (int ai = 0; ai < 2; ++ai)
#pragma unroll
                for (int m = 0; m < 4; ++m) {
                    const f32x4 a0 = acc[ai][0][m][0], a1 = acc[ai][0][m][1], a2 = acc[ai][1][m][0], a3 = acc[ai][1][m][1];
                    const f32x4 t0 = a0 * sl, t1 = a1 * sl, t2 = a2 * sl, t3 = a3 * sl;
#pragma unroll
                    for (int e = 0; e < 4; ++e) {
                        const f32x4 v = silu4(a0[e], a1[e], a2[e], a3[e], t0[e], t1[e], t2[e], t3[e], is);
                        asm volatile("global_store_dwordx4 %0, %1, %2 " OUT_ST "\n\ts_nop 1" :: "v"(oo), "v"(v), "s"(ob + (size_t)(ai * 128 + m * 16 + e) * N_) : "memory"); } }
        }
        if (wr == 1) BAR;
        STAMP(3 + 2 * ti);
        if (!has_next) break;
        cur = nxt; cA = nA; cB = nB; kta = KT_OF(cur); ++ti;
    }
    WAIT_V(0);
    STAMP(14);
    if (wr == 0) BAR;
}

void make_schedule(Params& p) {
    constexpr int G = NWG - NPULL, E = 1024 - 4 * G;
    static_assert(E >= 0 && E <= G && E <= 256 && NPULL <= ROLE_PACK2, "schedule");
    int ngx[8] = {}, nex[8] = {};
    int role[NWG]; unsigned short tile[NWG][MAXT];
    for (int b = 0; b < NWG; ++b) { role[b] = b >= G ? NWG - 1 - b : ROLE_GEMM; for (int i = 0; i < MAXT; ++i) tile[b][i] = 0xffff; }
    for (int b = 0; b < G; ++b) { ++ngx[b & 7]; if (b < E) ++nex[b & 7]; }
    for (int i = 0; i < NU - NWG; ++i) role[G - 1 - i] = ROLE_PACK2 + i;
    int sr = 0, sc = 0;
    if (E > 0) { int best = 1000; for (int a = 1; a <= 16; ++a) { const int b = (E + a - 1) / a; if (a + b <= best) { best = a + b; sr = a; sc = b; } } }
    signed char rnd[16][64]; int cnt = 0;
    for (int pn = 0; pn < 64; ++pn) for (int pm = 0; pm < 16; ++pm) rnd[pm][pn] = 100;
    for (int pn = 0; pn < sc; ++pn) for (int pm = 0; pm < sr; ++pm) if (cnt < E) { rnd[pm][pn] = -1; ++cnt; }
    cnt = 0;
    for (int pn = 0; pn < 64; ++pn) for (int pm = 0; pm < 16; ++pm) if (rnd[pm][pn] == 100) { rnd[pm][pn] = (signed char)(cnt / G); ++cnt; }
    int ntile[NWG] = {};
    for (int r = -1; r < 4; ++r) {
        int xcd = 0, slot = 0;
        for (int band = 0; band < 4; ++band)
            for (int c = 0; c < 64; ++c) { const int pn = (band & 1) ? 63 - c : c;
                for (int pm = 4 * band; pm < 4 * band + 4; ++pm) if (rnd[pm][pn] == r) {
                    while (slot >= (r < 0 ? nex[xcd] : ngx[xcd])) { ++xcd; slot = 0; }
                    const int b = slot * 8 + xcd; ++slot;
                    tile[b][ntile[b]++] = (unsigned short)(pm * 64 + pn);
                } }
    }
    bool used[NPANEL] = {}; int np = 0;
    for (int r = -1; r < 4; ++r) {
        p.seg[r + 1] = np;
        bool need[NPANEL] = {};
        for (int pm = 0; pm < 16; ++pm) for (int pn = 0; pn < 64; ++pn) if (rnd[pm][pn] == r) { need[pm] = true; need[16 + pn] = true; }
        int ix = 0, iw = 16;
        for (;;) {
            while (ix < 16 && (!need[ix] || used[ix])) ++ix;
            while (iw < NPANEL && (!need[iw] || used[iw])) ++iw;
            if (ix >= 16 && iw >= NPANEL) break;
            if (ix < 16) { used[ix] = true; p.panel[np++] = (unsigned char)ix; }
            if (iw < NPANEL) { used[iw] = true; p.panel[np++] = (unsigned char)iw; }
        }
    }
    p.seg[NSEG] = np;
    for (int i = 0; i < NPANEL; ++i) p.pidx[p.panel[i]] = (unsigned char)i;
    for (int b = 0; b < NWG; ++b) {
        p.rec[b][0] = tile[b][0] | (unsigned)tile[b][1] << 16; p.rec[b][1] = tile[b][2] | (unsigned)tile[b][3] << 16;
        p.rec[b][2] = tile[b][4] | (unsigned)(role[b] << 9 | unit_of(p, b, 0)) << 16;
    }
}

}

extern "C" void kernel_launch(int rank, int nranks, ncclComm_t comm,
                              hipStream_t stream,
                              void* const* d_in, const int* in_sizes, int n_in,
                              void* d_out, int out_size,
                              const int* in_dtypes, int out_dtype,
                              void* const* d_out_all,
                              void* const* d_ws_all, size_t ws_bytes) {
    (void)comm; (void)in_dtypes; (void)out_dtype; (void)d_out_all;
    if (nranks != NR || (unsigned)rank >= (unsigned)NR || n_in != 4 || in_sizes[0] != M_ * KSH || in_sizes[1] != KSH * N_ ||
        out_size != M_ * N_ || ws_bytes < WS_NEED) {
        fprintf(stderr, "kernel_launch: unexpected problem shape\n");
        return;
    }
    HIP_CHECK(hipFuncSetAttribute((const void*)fused_kernel, hipFuncAttributeMaxDynamicSharedMemorySize, LDS_BYTES));
    Params p;
    p.x = (const unsigned char*)d_in[0]; p.w = (const unsigned char*)d_in[1];
    p.sx = (const float*)d_in[2]; p.sw = (const float*)d_in[3];
    p.out = (float*)d_out;
    for (int i = 0; i < NR; ++i) p.ws[i] = (unsigned char*)d_ws_all[i];
    p.rank = rank;
    p.mode = 0;
    make_schedule(p);
    hipLaunchKernelGGL(fused_kernel, dim3(NWG), dim3(NTHR), LDS_BYTES, stream, p);
    HIP_CHECK(hipGetLastError());
}
```

```cpp
#include <hip/hip_runtime.h>
#include <rccl/rccl.h>
#include <cstdio>

#define HIP_CHECK(expr) do { hipError_t _e = (expr); if (_e != hipSuccess) { \
    fprintf(stderr, "HIP error %s at %s:%d\n", hipGetErrorString(_e), __FILE__, __LINE__); \
    return; } } while (0)

namespace {

#define LAS __attribute__((address_space(3)))
typedef int i32x4 __attribute__((ext_vector_type(4)));
typedef int i32x8 __attribute__((ext_vector_type(8)));
typedef unsigned u32x4 __attribute__((ext_vector_type(4)));
typedef float f32x4 __attribute__((ext_vector_type(4)));
typedef float f32x2 __attribute__((ext_vector_type(2)));

constexpr int M_ = 4096, N_ = 16384, NR = 8, KSH = 512;
constexpr int NWG = 256, NTHR = 512;
constexpr unsigned UNIT = 32768;
constexpr unsigned HTB = 16384;
constexpr int NPANEL = 80, NU = 4 * NPANEL;
constexpr unsigned SRC_BYTES = NU * UNIT;
constexpr unsigned PAR_BYTES = NR * SRC_BYTES;
constexpr unsigned WS_CTR = 0;
constexpr unsigned WS_TMO = 16384;
constexpr unsigned WS_TK = 20480;
constexpr unsigned WS_LF = 32768;
constexpr unsigned WS_SF = 65536;
constexpr unsigned WS_DBG = 512 * 1024;
constexpr unsigned WS_STAGE = 1u << 20;
constexpr size_t WS_NEED = (size_t)WS_STAGE + 2ull * PAR_BYTES;
constexpr unsigned long long DEADLINE_TICKS = 40000000ull;

#define WPP 3
#define PULL_W 10
#define PULL_G 2
#define TAILM 0
#define SEG2 1
#define MORD 1
#define PLAN_AHEAD 1
#define PULL_XCD_ORDER {1, 3, 7, 5, 2, 6, 0, 4}
#define PULL_XCD_HEAD 32
#define EARLY_EVEN 1
#define NSH 64
#define SH_INS 16
#define LAST_WT 0
#define STAGE_AUX 16
#define SPIN_SLEEP 4
#define OUT_ST "nt"
constexpr int NPULL = 7 * WPP;
constexpr int MAXT = 5, NSEG = 6;
constexpr int SFN = NU + 16;
constexpr int NSO = NSH / NR, CSH = 64 - NSH / 16;
static_assert(NSH % 16 == 0 && NSO <= 16 && NU + NSH <= 464, "shared tiles");
constexpr int LDS_CTL = 8 * HTB, LDS_BYTES = LDS_CTL + 2048;
constexpr int ROLE_GEMM = 127, ROLE_PACK2 = 32;

struct Params {
    const unsigned char* x; const unsigned char* w; const float* sx; const float* sw;
    float* out;
    unsigned char* ws[NR];
    const unsigned char* outs[NR];
    int rank; int mode;
    int seg[NSEG + 1];
    unsigned char panel[NPANEL];
    unsigned char pidx[NPANEL];
    unsigned rec[NWG][3];
};
static_assert(sizeof(Params) <= 4096, "kernel arguments");

__host__ __device__ __forceinline__ int unit_of(const Params& p, int o, int q) {
    int s0 = 0, n = 1;
#pragma unroll
    for (int s = 0; s < NSEG; ++s) if (o >= 4 * p.seg[s] && p.seg[s + 1] > p.seg[s]) { s0 = p.seg[s]; n = p.seg[s + 1] - s0; }
    o -= 4 * s0;
    const int j = (o >= n) + (o >= 2 * n) + (o >= 3 * n);
    int i = o - j * n + (q * n) / 7; if (i >= n) i -= n;
    return (int)p.panel[s0 + i] * 4 + j;
}

#define WAIT_V(n) asm volatile("s_waitcnt vmcnt(" #n ")" ::: "memory")
#define WAIT_VX(n) WAIT_V(n)
#define WAIT_L(n) asm volatile("s_waitcnt lgkmcnt(" #n ")" ::: "memory")
#define BAR __builtin_amdgcn_s_barrier()
#define FENCE asm volatile("" ::: "memory")
#define SCHED __builtin_amdgcn_sched_barrier(0)
#define STAMP(i) do { } while (0)

__device__ __forceinline__ void st16_sys(void* p, u32x4 v) { asm volatile("global_store_dwordx4 %0, %1, off sc0 sc1\n\ts_nop 1" :: "v"(p), "v"(v) : "memory"); }
__device__ __forceinline__ void ld4_dev_async(unsigned& v, const void* base, unsigned off) { asm volatile("global_load_dword %0, %1, %2 sc1" : "+v"(v) : "v"(off), "s"(base) : "memory"); }
__device__ __forceinline__ unsigned ld4_sys_now(const void* p) { unsigned v; asm volatile("global_load_dword %0, %1, off sc0 sc1\n\ts_waitcnt vmcnt(0)" : "=&v"(v) : "v"(p) : "memory"); return v; }
__device__ __forceinline__ unsigned ld4_dev_now(const void* p) { unsigned v; asm volatile("global_load_dword %0, %1, off sc1\n\ts_waitcnt vmcnt(0)" : "=&v"(v) : "v"(p) : "memory"); return v; }

__device__ __forceinline__ bool give_up(unsigned long long t0, unsigned call, unsigned* tmo) {
    if ((unsigned)__builtin_amdgcn_readfirstlane(ld4_dev_now(tmo)) == call) return true;
    if (__builtin_amdgcn_s_memrealtime() - t0 > DEADLINE_TICKS) {
        __hip_atomic_store(tmo, call, __ATOMIC_RELAXED, __HIP_MEMORY_SCOPE_AGENT);
        WAIT_V(0);
        return true;
    }
    return false;
}
__device__ __noinline__ void wait_flags(const unsigned* flag, unsigned call, unsigned* tmo) {
    const unsigned long long t0 = __builtin_amdgcn_s_memrealtime();
    for (unsigned it = 1;; ++it) {
        if (__all(ld4_dev_now(flag) >= call)) break;
        if ((it & 7u) == 0u && give_up(t0, call, tmo)) break;
        __builtin_amdgcn_s_sleep(12);
    }
}

__device__ __forceinline__ unsigned lf_index(int u, int src) {
    return (unsigned)(u >> 2) * 32u + (unsigned)(src * 4 + (u & 3));
}

__device__ __forceinline__ void pack_load(const Params& p, int u, u32x4 (&o)[4], int wid, int lane) {
    const int j = u & 3, pa = u >> 2;
    if (pa < 16) {
#pragma unroll
        for (int i = 0; i < 4; ++i) { const int pc = wid * 4 + i;
            const int row = 256 * pa + 16 * (pc >> 1) + (lane & 15), col = 128 * j + 64 * (pc & 1) + 16 * (lane >> 4);
            o[i] = *(const u32x4*)(p.x + (size_t)row * KSH + col); }
    } else {
        const unsigned char* src = p.w + (size_t)(128 * j + 16 * wid) * N_ + 256 * (pa - 16) + 4 * lane;
        unsigned r[16];
#pragma unroll
        for (int i = 0; i < 16; ++i) r[i] = *(const unsigned*)(src + (size_t)i * N_);
#pragma unroll
        for (int q = 0; q < 4; ++q) {
            const unsigned a = r[4 * q], b = r[4 * q + 1], c = r[4 * q + 2], d = r[4 * q + 3];
            const unsigned ab_lo = __builtin_amdgcn_perm(b, a, 0x05010400u), ab_hi = __builtin_amdgcn_perm(b, a, 0x07030602u);
            const unsigned cd_lo = __builtin_amdgcn_perm(d, c, 0x05010400u), cd_hi = __builtin_amdgcn_perm(d, c, 0x07030602u);
            o[0][q] = __builtin_amdgcn_perm(cd_lo, ab_lo, 0x05040100u);
            o[1][q] = __builtin_amdgcn_perm(cd_lo, ab_lo, 0x07060302u);
            o[2][q] = __builtin_amdgcn_perm(cd_hi, ab_hi, 0x05040100u);
            o[3][q] = __builtin_amdgcn_perm(cd_hi, ab_hi, 0x07060302u);
        }
    }
}
__device__ __forceinline__ void pack_store(int u, unsigned char* dst, const u32x4 (&o)[4], int wid, int lane) {
    const bool isx = (u >> 2) < 16;
    const unsigned base = isx ? (unsigned)wid * 4096u + (unsigned)lane * 16u
                              : (unsigned)(lane >> 4) * 4096u + (unsigned)(wid >> 2) * 1024u + (unsigned)((wid & 3) * 16 + (lane & 15)) * 16u;
#pragma unroll
    for (int c = 0; c < 4; ++c) st16_sys(dst + base + (isx ? c * 1024u : (c >> 1) * HTB + (c & 1) * 2048u), o[c]);
}

__device__ __forceinline__ float exp_clip(float t) {
    constexpr float LIM = 60.0f * 1.44269504088896340736f;
    return __builtin_amdgcn_exp2f(__builtin_amdgcn_fmed3f(t, -LIM, LIM));
}
__device__ __forceinline__ f32x4 silu4(float a0, float a1, float a2, float a3, float t0, float t1, float t2, float t3, float is) {
    const f32x2 e01 = {exp_clip(t0), exp_clip(t1)}, e23 = {exp_clip(t2), exp_clip(t3)};
    const f32x2 d01 = __builtin_elementwise_fma(e01, (f32x2){is, is}, (f32x2){is, is}), d23 = __builtin_elementwise_fma(e23, (f32x2){is, is}, (f32x2){is, is});
    float r0, r1, r2, r3;
    asm("v_rcp_f32 %0, %4\n\tv_rcp_f32 %1, %5\n\tv_rcp_f32 %2, %6\n\tv_rcp_f32 %3, %7\n\t"
        "v_mul_f32 %0, %8, %0\n\tv_mul_f32 %1, %9, %1\n\tv_mul_f32 %2, %10, %2\n\tv_mul_f32 %3, %11, %3"
        : "=&v"(r0), "=&v"(r1), "=&v"(r2), "=&v"(r3) : "v"(d01.x), "v"(d01.y), "v"(d23.x), "v"(d23.y), "v"(a0), "v"(a1), "v"(a2), "v"(a3));
    return (f32x4){r0, r1, r2, r3};
}

template <bool B> struct BoolC { static constexpr bool value = B; };

__global__ __launch_bounds__(NTHR, 2) void fused_kernel(const Params p) {
    extern __shared__ __attribute__((aligned(16))) unsigned char shm[];
    LAS unsigned char* lds = (LAS unsigned char*)shm;
    LAS unsigned* ctl = (LAS unsigned*)(lds + LDS_CTL);
    const int tid = threadIdx.x, wid = __builtin_amdgcn_readfirstlane(tid >> 6), lane = tid & 63;
    const int bid = blockIdx.x;
    const int rank = p.rank;
    unsigned char* ws = p.ws[rank];
    unsigned* tmo = (unsigned*)(ws + WS_TMO);
    STAMP(0);

    unsigned ticket = 0;
    if (tid == 0) ticket = __hip_atomic_fetch_add((unsigned*)(ws + WS_CTR + bid * 64), 1u, __ATOMIC_RELAXED, __HIP_MEMORY_SCOPE_AGENT);
    const unsigned rec0 = p.rec[bid][0], rec1 = p.rec[bid][1], rec2 = p.rec[bid][2];
    const int role = (int)(rec2 >> 25);
    int pu = (int)((rec2 >> 16) & 511u);
    u32x4 pk[4];
    if (role >= NPULL) pack_load(p, pu, pk, wid, lane);
    else if (tid < NU + NSH) {
        const int nin = 4 * p.seg[NSEG - 1];
        ((LAS unsigned short*)(ctl + 48))[tid] = (unsigned short)(tid < nin ? unit_of(p, tid, role % 7) : 512 + tid - nin);
    } else if (tid >= 464 + 2) ctl[tid - 464] = 0u;
    if (tid == 0) { ctl[0] = ticket + 1u; ctl[1] = 0u; }
    __syncthreads();
    const unsigned call = __builtin_amdgcn_readfirstlane(ctl[0]);
    const unsigned stage_off = WS_STAGE + (call & 1u) * PAR_BYTES;
    unsigned char* stage = ws + stage_off;

    if (!(p.mode & 1) && role >= NPULL) {
        for (int rep = 0;; ++rep) {
            pack_store(pu, stage + (size_t)rank * SRC_BYTES + (size_t)pu * UNIT, pk, wid, lane);
            WAIT_V(0); BAR; FENCE;
            if (tid < NR) {
                if (tid == rank) __hip_atomic_store((unsigned*)(ws + WS_LF) + lf_index(pu, rank), call, __ATOMIC_RELAXED, __HIP_MEMORY_SCOPE_AGENT);
                else __hip_atomic_store((unsigned*)(p.ws[tid] + WS_SF + (unsigned)(rank * SFN + pu) * 64u), call, __ATOMIC_RELAXED, __HIP_MEMORY_SCOPE_SYSTEM);
            }
            if (rep == 1 || role == ROLE_GEMM) break;
            pu = __builtin_amdgcn_readfirstlane(unit_of(p, NWG - NPULL + role - ROLE_PACK2, 0));
            pack_load(p, pu, pk, wid, lane);
        }
    }
    STAMP(1);

    if (role < NPULL) {
        if (p.mode & 2) return;
        const int q = role % 7, wq = role / 7, src = (rank + 1 + q) & (NR - 1);
        constexpr int C_PLAN = 2, C_ISSUED = 4, C_LANDED = 6, C_FREED = 8, C_DESC = 16, C_CNT = 32, C_TAB = 48;
        LAS unsigned short* utab = (LAS unsigned short*)(ctl + C_TAB);
        const unsigned cbase = (unsigned)(size_t)ctl;
#define CGET(w) ({ int v_; asm volatile("ds_read_b32 %0, %1\n\ts_waitcnt lgkmcnt(0)" : "=&v"(v_) : "v"(cbase + (unsigned)(w) * 4u) : "memory"); __builtin_amdgcn_readfirstlane(v_); })
#define CPUT(w, v) asm volatile("ds_write_b32 %0, %1" :: "v"(cbase + (unsigned)(w) * 4u), "v"((int)(v)) : "memory")
        const unsigned long long tstart = __builtin_amdgcn_s_memrealtime();
#define SPIN_GUARD() do { __builtin_amdgcn_s_sleep(SPIN_SLEEP); if (__builtin_amdgcn_s_memrealtime() - tstart > 2 * DEADLINE_TICKS) { \
            __hip_atomic_store(tmo, call, __ATOMIC_RELAXED, __HIP_MEMORY_SCOPE_AGENT); return; } } while (0)
#define SH_OFF(z) ({ const int i_ = src * NSO + ((z) >> 3); ((size_t)((i_ & 15) * 256 + ((z) & 7) * 32) * N_ + (size_t)(CSH + (i_ >> 4)) * 256) * 4; })
        const int npos = 4 * p.seg[NSEG - 1] + NSH;
        if (wid < 2) {
            const unsigned char* rbase = p.ws[src] + stage_off + (size_t)src * SRC_BYTES + (unsigned)wid * HTB;
            const unsigned loff = (unsigned)lane * 16u;
            int landed = 0;
            for (int k = 0;; ++k) {
                for (int spin = 0;; ++spin) {
                    const int f0 = CGET(C_FREED + 2 * wid), f1 = CGET(C_FREED + 2 * wid + 1);
                    if (CGET(C_PLAN) > k && f0 + 4 > k && f1 + 4 > k) break;
                    if (landed < k && spin >= 16) { WAIT_V(0); landed = k; CPUT(C_LANDED + wid, k); } else SPIN_GUARD();
                }
                const int u = CGET(C_DESC + (k & 15));
                if (u < 0) { WAIT_V(0); CPUT(C_LANDED + wid, k); break; }
                const unsigned char* g = u < 512 ? rbase + (size_t)u * UNIT : p.outs[src] + SH_OFF(u - 512) + (size_t)wid * (16u * N_ * 4u);
                const unsigned gs = u < 512 ? 1024u : N_ * 4u;
                CPUT(C_ISSUED + wid, k + 1);
                static_assert(PULL_W % PULL_G == 0 && PULL_W <= 16 - PULL_G, "window");
#pragma unroll
                for (int i0 = 0; i0 < 16; i0 += PULL_G) {
                    WAIT_VX(PULL_W);
                    if (i0 == PULL_W && landed < k) { landed = k; CPUT(C_LANDED + wid, k); }
#pragma unroll
                    for (int i = i0; i < i0 + PULL_G; ++i)
                        __builtin_amdgcn_global_load_lds((const unsigned*)(g + i * gs + loff), (LAS unsigned*)(lds + (k & 3) * UNIT + wid * HTB + i * 1024), 16, 0, 17  );
                }
            }
            STAMP(2);
        } else if (wid == 2) {
            unsigned* tk = (unsigned*)(ws + WS_TK + q * 128);
            const unsigned tk_base = (call - 1u) * (unsigned)(npos - 2 * WPP);
            int horizon = 0;
            for (int k = 0;; ++k) {
                while (k > CGET(C_ISSUED) + PLAN_AHEAD || k > CGET(C_ISSUED + 1) + PLAN_AHEAD) SPIN_GUARD();
                int pos;
                if (k < 3) pos = k * WPP + wq;
                else { unsigned t = 0; if (lane == 0) t = __hip_atomic_fetch_add(tk, 1u, __ATOMIC_RELAXED, __HIP_MEMORY_SCOPE_AGENT);
                    const unsigned r = (unsigned)__builtin_amdgcn_readfirstlane(t) - tk_base;
                    pos = r < (unsigned)(npos - 3 * WPP) ? 3 * WPP + (int)r : -1; }
                if (pos >= horizon) {
                    const unsigned long long t0 = __builtin_amdgcn_s_memrealtime();
                    horizon = pos;
                    for (;;) {
                        const int v = utab[horizon + lane < npos ? horizon + lane : npos - 1];
                        const unsigned long long nr = ~__ballot(ld4_sys_now(ws + WS_SF + (unsigned)(src * SFN + (v < 512 ? v : NU + ((v - 512) >> 3))) * 64u) >= call);
                        horizon += nr ? __builtin_ctzll(nr) : 64;
                        if (horizon > pos) break;
                        if (give_up(t0, call, tmo)) {
                            const int w = utab[pos];
                            if (w >= 512) { if (lane == 0) *(float*)((unsigned char*)p.out + SH_OFF(w - 512)) = __builtin_nanf(""); pos = -1; }
                            horizon = pos + 1; break; }
                        __builtin_amdgcn_s_sleep(2);
                    }
                }
                CPUT(C_DESC + (k & 15), pos >= 0 ? (int)utab[pos] : -1);
                CPUT(C_PLAN, k + 1);
                if (pos < 0) break;
            }
        } else if (wid >= 4) {
            const int i4 = wid - 4;
            const unsigned toff = (unsigned)i4 * 8192u + (unsigned)lane * 16u;
            unsigned char* lbase = stage + (size_t)src * SRC_BYTES + toff;
            int pend = -1, pend_u = 0;
#define ARRIVE() do { if (lane == 0) { unsigned old_; asm volatile("ds_add_rtn_u32 %0, %1, %2\n\ts_waitcnt lgkmcnt(0)" : "=&v"(old_) : "v"(cbase + (unsigned)(C_CNT + (pend & 15)) * 4u), "v"(1u) : "memory"); \
                if ((old_ & 3u) == 3u && pend_u < 512) { __hip_atomic_store((unsigned*)(ws + WS_LF) + lf_index(pend_u, src), call, __ATOMIC_RELAXED, __HIP_MEMORY_SCOPE_AGENT); PUBSTAMP(); } } } while (0)
#define PUBSTAMP() do { } while (0)
            for (int k = 0;; ++k) {
                bool end = false;
                while (CGET(C_LANDED + (i4 >> 1)) <= k) {
                    if (pend >= 0) { WAIT_V(0); ARRIVE(); pend = -1; }
                    else if (CGET(C_PLAN) > k && CGET(C_DESC + (k & 15)) < 0) { end = true; break; }
                    else SPIN_GUARD();
                }
                if (end) break;
                const int u = CGET(C_DESC + (k & 15));
                unsigned char* dst = u < 512 ? lbase + (size_t)u * UNIT : (unsigned char*)p.out + SH_OFF(u - 512) + (size_t)i4 * (8u * N_ * 4u) + lane * 16;
                const unsigned ds = u < 512 ? 1024u : N_ * 4u;
                const unsigned la = (unsigned)(size_t)(lds + (k & 3) * UNIT + toff);
#pragma unroll
                for (int h = 0; h < 2; ++h) { u32x4 t0, t1, t2, t3;
                    asm volatile("ds_read_b128 %0, %4\n\tds_read_b128 %1, %4 offset:1024\n\tds_read_b128 %2, %4 offset:2048\n\tds_read_b128 %3, %4 offset:3072\n\t"
                                 "s_waitcnt lgkmcnt(3)\n\tglobal_store_dwordx4 %5, %0, off sc1\n\t"
                                 "s_waitcnt lgkmcnt(2)\n\tglobal_store_dwordx4 %6, %1, off sc1\n\t"
                                 "s_waitcnt lgkmcnt(1)\n\tglobal_store_dwordx4 %7, %2, off sc1\n\t"
                                 "s_waitcnt lgkmcnt(0)\n\tglobal_store_dwordx4 %8, %3, off sc1\n\ts_nop 1"
                                 : "=&v"(t0), "=&v"(t1), "=&v"(t2), "=&v"(t3)
                                 : "v"(la + h * 4096), "v"(dst + (4 * h) * ds), "v"(dst + (4 * h + 1) * ds), "v"(dst + (4 * h + 2) * ds), "v"(dst + (4 * h + 3) * ds) : "memory"); }
                CPUT(C_FREED + i4, k + 1);
                if (pend >= 0) { WAIT_V(8); ARRIVE(); }
                pend = k; pend_u = u;
            }
        }
        return;
    }

    if (p.mode & 4) return;
    const int wr = wid >> 2, wc = wid & 3;
    int tdesc = -1;
    { const unsigned rw = lane < 2 ? rec0 : (lane < 4 ? rec1 : rec2), t = lane < MAXT ? (rw >> (16 * (lane & 1))) & 0xffffu : 0xffffu;
      if (t != 0xffffu) {
          const int ia = p.pidx[t >> 6], ib = p.pidx[16 + (t & 63u)], hi = ia > ib ? ia : ib, lo = ia > ib ? ib : ia;
          int s0 = 0, n = 1;
#pragma unroll
          for (int s = 0; s < NSEG; ++s) if (hi >= p.seg[s] && p.seg[s + 1] > p.seg[s]) { s0 = p.seg[s]; n = p.seg[s + 1] - s0; }
          int key[7];
#pragma unroll
          for (int q = 0; q < 7; ++q) { const int r = (q * n) / 7; int kh = hi - s0 - r; if (kh < 0) kh += n; int kl = lo - s0 - r; if (kl < 0) kl += n;
              key[q] = ((lo >= s0 && kl > kh) ? kl : kh) * 8 + q; }
          unsigned w = 0;
#pragma unroll
          for (int q = 0; q < 7; ++q) { int rk = 0;
#pragma unroll
              for (int r = 0; r < 7; ++r) rk += key[r] < key[q];
              w |= (unsigned)q << (3 * rk); }
          tdesc = (int)(((t >> 6) << 27) | ((t & 63u) << 21) | w);
      } }
    const int ks = lane & 31, kj = ks < 4 ? ks : (ks - 4) / 7, ksh = ks < 4 ? 0 : 3 * ((ks - 4) - kj * 7);
#define KT_OF(d) (ks < 4 ? rank * 4 + ks : ((rank + 1 + (((d) >> ksh) & 7)) & 7) * 4 + kj)
#define PM_OF(d) (((d) >> 27) & 15)
#define PN_OF(d) (((d) >> 21) & 63)
#define FOFF_OF(d, kt) ((unsigned)(((lane < 32) ? PM_OF(d) : 16 + PN_OF(d)) * 32 + (kt)) * 4u)
    const unsigned* lf = (const unsigned*)(ws + WS_LF);
    const unsigned voff = (unsigned)tid * 16u;
    const unsigned ldsw = (unsigned)wid * 1024u;
    const LAS unsigned char* la = lds + wr * 8192 + lane * 16;
    const LAS unsigned char* lb = lds + 4 * HTB + wc * 4096 + lane * 16;
#define SA(b, h) (((b) * 2 + (h)) * HTB)
#define SB(b, h) ((4 + (b) * 2 + (h)) * HTB)
#define ABL_NST 2
#define STAGE(bufoff, gbase) do { _Pragma("unroll") for (int _i = 0; _i < ABL_NST; ++_i) \
        __builtin_amdgcn_global_load_lds((const unsigned*)((gbase) + _i * 8192 + voff), (LAS unsigned*)(lds + (bufoff) + ldsw + _i * 8192), 16, 0, STAGE_AUX); } while (0)
#define LDFRAG(ptr) __builtin_shufflevector(*(const LAS i32x4*)(ptr), *(const LAS i32x4*)((ptr) + 1024), 0, 1, 2, 3, 4, 5, 6, 7)
#define ABL_NLA 4
#define LDA(dst, b, h) do { _Pragma("unroll") for (int m = 0; m < ABL_NLA; ++m) dst[m] = LDFRAG(la + SA(b, h) + m * 2048); } while (0)
#define LDB(dst, b, h) do { _Pragma("unroll") for (int n = 0; n < 2; ++n) dst[n] = LDFRAG(lb + SA(b, h) + n * 2048); } while (0)
#define MFMA1(ai, bj, At, Bt, Z, i) do { constexpr int m = (i) >> 1, n = (i) & 1; \
        if constexpr (Z) asm volatile("v_mfma_f32_16x16x128_f8f6f4 %0, %1, %2, 0 cbsz:1 blgp:1" : "+v"(acc[ai][bj][m][n]) : "v"(At[m]), "v"(Bt[n])); \
        else acc[ai][bj][m][n] = __builtin_amdgcn_mfma_scale_f32_16x16x128_f8f6f4(At[m], Bt[n], acc[ai][bj][m][n], 1, 1, 0, 0, 0, 0); } while (0)
#define MMA(ai, bj, At, Bt, Z, B) do { __builtin_amdgcn_s_setprio(1); \
        MFMA1(ai, bj, At, Bt, Z, 0); MFMA1(ai, bj, At, Bt, Z, 1); MFMA1(ai, bj, At, Bt, Z, 2); MFMA1(ai, bj, At, Bt, Z, 3); MFMA1(ai, bj, At, Bt, Z, 4); \
        if (TAILM == 3) { __builtin_amdgcn_s_setprio(2); SCHED; B; SCHED; } \
        MFMA1(ai, bj, At, Bt, Z, 5); \
        if (TAILM == 2) { __builtin_amdgcn_s_setprio(2); SCHED; B; SCHED; } \
        MFMA1(ai, bj, At, Bt, Z, 6); \
        if (TAILM == 1) { __builtin_amdgcn_s_setprio(2); SCHED; B; SCHED; } \
        MFMA1(ai, bj, At, Bt, Z, 7); \
        __builtin_amdgcn_s_setprio(0); \
        if (TAILM == 0) { B; } \
        SCHED; } while (0)
#define KT(s) __builtin_amdgcn_readlane(kta, (s))
#define KOFF(kt) ((unsigned)((kt) >> 2) * SRC_BYTES + (unsigned)((kt) & 3) * UNIT)
#define MASK_OF(f) ({ const unsigned long long b_ = __ballot((f) >= call); (unsigned)b_ & (unsigned)(b_ >> 32); })
#define NEED(d, s, kt) do { if (!((mask >> (s)) & 1u)) wait_flags(lf + ((lane & 1) ? PM_OF(d) : 16 + PN_OF(d)) * 32 + (kt), call, tmo); } while (0)

    int ti = 0, cur = __builtin_amdgcn_readlane(tdesc, 0);
    if (cur < 0) return;
    f32x4 acc[2][2][4][2] = {};
    i32x8 At[4] = {}, B0[2], B1[2];
    const unsigned char* cA = stage + (size_t)PM_OF(cur) * 4 * UNIT;
    const unsigned char* cB = stage + (size_t)(16 + PN_OF(cur)) * 4 * UNIT;
    int kta = KT_OF(cur);
    unsigned foff = FOFF_OF(cur, kta), mask = 0, fa = 0, fb = 0, tv = 0;
    const float scale = __builtin_bit_cast(float, __builtin_amdgcn_readfirstlane(__builtin_bit_cast(int, p.sx[0] * p.sw[0])));
    const float sl = -scale * 1.44269504088896340736f, inv_s = 1.0f / scale;
    NEED(cur, 0, rank * 4); NEED(cur, 1, rank * 4 + 1);
    { const unsigned k0 = KOFF(rank * 4), k1 = k0 + UNIT;
      ld4_dev_async(fb, lf, foff);
      STAGE(SB(0, 0), cB + k0); STAGE(SA(0, 0), cA + k0); STAGE(SB(0, 1), cB + k0 + HTB); STAGE(SA(0, 1), cA + k0 + HTB);
      STAGE(SB(1, 0), cB + k1); STAGE(SA(1, 0), cA + k1); STAGE(SB(1, 1), cB + k1 + HTB);
      WAIT_V(0); BAR;
      if (wr == 1) BAR; }
    int pub = -1;
    bool rb = true;
    for (;;) {
        int nxt = __builtin_amdgcn_readlane(tdesc, ti + 1);
        const bool has_next = nxt >= 0;
        if (!has_next) nxt = cur;
        const unsigned char* nA = stage + (size_t)PM_OF(nxt) * 4 * UNIT;
        const unsigned char* nB = stage + (size_t)(16 + PN_OF(nxt)) * 4 * UNIT;
        auto trip = [&](auto first_c, const int t) __attribute__((always_inline)) {
            constexpr bool first = decltype(first_c)::value;
#define WAIT_K() do { if constexpr (first) WAIT_V(42); else WAIT_V(10); } while (0)
            const bool last = (t == 30);
            const int s2 = (t + 2) & 31, s3 = (t + 3) & 31;
            const int kt2 = KT(s2), kt3 = KT(s3);
            const unsigned k2 = KOFF(kt2), k3 = KOFF(kt3);
            const unsigned char* a1 = cA + KOFF(KT(t + 1));
            const unsigned char* a2 = (last ? nA : cA) + k2; const unsigned char* b2 = (last ? nB : cB) + k2;
            const unsigned char* a3 = (last ? nA : cA) + k3; const unsigned char* b3 = (last ? nB : cB) + k3;
            const int qd = last ? nxt : cur;
#define WAIT_S() do { if constexpr (first) WAIT_V(40); else WAIT_V(8); } while (0)
#define MMA2(ai, Z, B) do { __builtin_amdgcn_s_setprio(1); \
            MFMA1(ai, 0, At, B0, Z, 0); MFMA1(ai, 0, At, B0, Z, 1); MFMA1(ai, 1, At, B1, Z, 0); MFMA1(ai, 1, At, B1, Z, 1); \
            MFMA1(ai, 1, At, B1, Z, 3); MFMA1(ai, 1, At, B1, Z, 2); MFMA1(ai, 0, At, B0, Z, 3); MFMA1(ai, 0, At, B0, Z, 2); \
            MFMA1(ai, 0, At, B0, Z, 4); MFMA1(ai, 0, At, B0, Z, 5); MFMA1(ai, 1, At, B1, Z, 4); MFMA1(ai, 1, At, B1, Z, 5); \
            MFMA1(ai, 1, At, B1, Z, 7); MFMA1(ai, 1, At, B1, Z, 6); MFMA1(ai, 0, At, B0, Z, 7); MFMA1(ai, 0, At, B0, Z, 6); \
            __builtin_amdgcn_s_setprio(0); B; SCHED; } while (0)
            if (last) ld4_dev_async(tv, tmo, 0u);
            LDB(B0, 0, 0); LDA(At, 0, 0); LDB(B1, 0, 1); STAGE(SA(1, 1), a1 + HTB);
            WAIT_S();
            if (rb) { asm volatile("" : "+v"(fb)); mask = MASK_OF(fb); }
            WAIT_L(0); BAR; MMA2(0, first, BAR);
            NEED(qd, s2, kt2);
            rb = (mask != 0xffffffffu);
            if (rb) ld4_dev_async(fb, lf, foff);
            LDA(At, 0, 1); STAGE(SB(0, 0), b2); STAGE(SA(0, 0), a2); STAGE(SB(0, 1), b2 + HTB);
            WAIT_S(); WAIT_L(0); BAR; MMA2(1, first, BAR);
            LDB(B0, 1, 0); LDA(At, 1, 0); LDB(B1, 1, 1); STAGE(SA(0, 1), a2 + HTB);
            WAIT_V(8);
            if constexpr (first) { if (pub >= 0) {
                unsigned old = 0;
                if (lane == 0) asm volatile("ds_add_rtn_u32 %0, %1, %2\n\ts_waitcnt lgkmcnt(0)" : "=&v"(old) : "v"((unsigned)(size_t)(ctl + 1)), "v"(1u) : "memory");
                if ((__builtin_amdgcn_readfirstlane(old) & 7) == 7 && lane < NR && lane != rank)
                    __hip_atomic_store((unsigned*)(p.ws[lane] + WS_SF + (unsigned)(rank * SFN + NU + pub) * 64u), call, __ATOMIC_RELAXED, __HIP_MEMORY_SCOPE_SYSTEM);
                pub = -1; } }
            if (rb) { asm volatile("" : "+v"(fb)); mask = MASK_OF(fb); }
            WAIT_L(0); BAR; MMA2(0, false, BAR);
            NEED(qd, s3, kt3);
            const bool sw = (t == 28);
            if (sw) foff = FOFF_OF(nxt, KT_OF(nxt));
            rb = sw || (mask != 0xffffffffu);
            if (rb) ld4_dev_async(fb, lf, foff);
            LDA(At, 1, 1); STAGE(SB(1, 0), b3); STAGE(SA(1, 0), a3); STAGE(SB(1, 1), b3 + HTB);
            WAIT_V(8);
            asm volatile("" : "+v"(tv));
            WAIT_L(0); BAR; MMA2(1, false, if (!(last && wr == 1)) BAR);
        };
        trip(BoolC<true>{}, 0);
        for (int t = 2; t < 32; t += 2) trip(BoolC<false>{}, t);
        STAMP(2 + 2 * ti);
        {
            const float is = ((unsigned)__builtin_amdgcn_readfirstlane(tv) == call) ? __builtin_nanf("") : inv_s;
            const float* ob = p.out + (size_t)(PM_OF(cur) * 256 + wr * 64) * N_ + PN_OF(cur) * 256 + wc * 64;
            const unsigned oo = (unsigned)(4 * (lane >> 4)) * (N_ * 4u) + (unsigned)(lane & 15) * 16u;
            auto epi = [&](auto shared_c, const unsigned oo) __attribute__((always_inline)) {
#pragma unroll
                for (int ai = 0; ai < 2; ++ai)
#pragma unroll
                    for (int m = 0; m < 4; ++m) {
                        const f32x4 a0 = acc[ai][0][m][0], a1 = acc[ai][0][m][1], a2 = acc[ai][1][m][0], a3 = acc[ai][1][m][1];
                        const f32x4 t0 = a0 * sl, t1 = a1 * sl, t2 = a2 * sl, t3 = a3 * sl;
#pragma unroll
                        for (int e = 0; e < 4; ++e) {
                            const f32x4 v = silu4(a0[e], a1[e], a2[e], a3[e], t0[e], t1[e], t2[e], t3[e], is);
                            const float* rowp = ob + (size_t)(ai * 128 + m * 16 + e) * N_;
                            if constexpr (decltype(shared_c)::value) asm volatile("global_store_dwordx4 %0, %1, %2 sc0 sc1\n\ts_nop 1" :: "v"(oo), "v"(v), "s"(rowp) : "memory");
                            else asm volatile("global_store_dwordx4 %0, %1, %2 " OUT_ST "\n\ts_nop 1" :: "v"(oo), "v"(v), "s"(rowp) : "memory"); } }
            };
            if (NSH > 0 && PN_OF(cur) >= CSH) pub = (PN_OF(cur) - CSH) * 16 + PM_OF(cur) - rank * NSO;
            if (pub >= 0 || (LAST_WT && !has_next)) epi(BoolC<true>{}, oo);
            else epi(BoolC<false>{}, oo);
        }
        if (wr == 1) BAR;
        STAMP(3 + 2 * ti);
        if (!has_next) break;
        cur = nxt; cA = nA; cB = nB; kta = KT_OF(cur); ++ti;
    }
    WAIT_V(0);
    STAMP(14);
    if (wr == 0) BAR;
}

void make_schedule(Params& p) {
    constexpr int G = NWG - NPULL, E = 1024 - (NSH - NSO) - 4 * G;
    static_assert(E >= 0 && E <= G && E <= 256 && NPULL <= ROLE_PACK2 && ROLE_PACK2 + NU - G <= ROLE_GEMM, "schedule");
    int ngx[8] = {}, nex[8] = {};
    int role[NWG]; unsigned short tile[NWG][MAXT];
    int npx[8] = {}, ppos[NWG];
    { const int order[8] = PULL_XCD_ORDER; for (int i = 0; i < NPULL; ++i) ++npx[order[i < PULL_XCD_HEAD ? i % 4 : (i - PULL_XCD_HEAD) % 8]]; }
    for (int b = 0; b < NWG; ++b) { role[b] = ROLE_GEMM; ppos[b] = 0; for (int i = 0; i < MAXT; ++i) tile[b][i] = 0xffff; }
    { int c = 0; for (int b = NWG - 1; b >= 0; --b) if ((b >> 3) >= 32 - npx[b & 7]) role[b] = c++; }
    for (int x = 0; x < 8; ++x) ngx[x] = 32 - npx[x];
    { int left = E; for (int pass = 0; pass < 2; ++pass) for (bool any = true; any && left > 0;) { any = false;
        for (int x = pass; x < 8 && left > 0; x += 2) if (nex[x] < ngx[x]) { ++nex[x]; --left; any = true; } } }
    { int c = 0; for (int b = 0; b < NWG; ++b) if (role[b] == ROLE_GEMM) ppos[b] = c++;
      c = 0; for (int b = NWG - 1; b >= 0 && c < NU - G; --b) if (role[b] == ROLE_GEMM) role[b] = ROLE_PACK2 + c++; }
    int sr = 0, sc = 0;
    if (E > 0) { int best = 1000; for (int a = 1; a <= 16; ++a) { const int b = (E + a - 1) / a; if (a + b <= best) { best = a + b; sr = a; sc = b; } } }
    int col[64];
    for (int c = 0; c < 64; ++c) col[c] = c < SH_INS ? c : (c < SH_INS + 64 - CSH ? CSH + c - SH_INS : c - (64 - CSH));
    signed char rnd[16][64]; int cnt = 0;
    for (int pn = 0; pn < 64; ++pn) for (int pm = 0; pm < 16; ++pm)
        rnd[pm][pn] = (pn >= CSH && ((pn - CSH) * 16 + pm) / (NSO ? NSO : 1) != p.rank) ? 99 : 100;
    for (int pn = 0; pn < sc; ++pn) for (int pm = 0; pm < sr; ++pm) if (cnt < E) { rnd[pm][pn] = -1; ++cnt; }
    cnt = 0;
    for (int c = 0; c < 64; ++c) for (int pm = 0; pm < 16; ++pm) if (rnd[pm][col[c]] == 100) { rnd[pm][col[c]] = (signed char)(cnt / G); ++cnt; }
    int ntile[NWG] = {};
    for (int r = -1; r < 4; ++r) {
        int xcd = 0, slot = 0;
        for (int band = 0; band < 4; ++band)
            for (int c = 0; c < 64; ++c) { const int pn = col[(band & 1) ? 63 - c : c];
                for (int pm = 4 * band; pm < 4 * band + 4; ++pm) if (rnd[pm][pn] == r) {
                    while (slot >= (r < 0 ? nex[xcd] : ngx[xcd])) { ++xcd; slot = 0; }
                    const int b = slot * 8 + xcd; ++slot;
                    tile[b][ntile[b]++] = (unsigned short)(pm * 64 + pn);
                } }
    }
    bool used[NPANEL] = {}; int np = 0;
    for (int r = -1; r < 4; ++r) {
        p.seg[r + 1] = np;
        bool need[NPANEL] = {};
        for (int pm = 0; pm < 16; ++pm) for (int pn = 0; pn < 64; ++pn) if (rnd[pm][pn] == r) { need[pm] = true; need[16 + pn] = true; }
        int ix = 0, iw = 16;
        for (;;) {
            while (ix < 16 && (!need[ix] || used[ix])) ++ix;
            while (iw < NPANEL && (!need[iw] || used[iw])) ++iw;
            if (ix >= 16 && iw >= NPANEL) break;
            if (ix < 16) { used[ix] = true; p.panel[np++] = (unsigned char)ix; }
            if (iw < NPANEL) { used[iw] = true; p.panel[np++] = (unsigned char)iw; }
        }
    }
    p.seg[NSEG - 1] = np;
    for (int i = 0; i < NPANEL; ++i) if (!used[i]) p.panel[np++] = (unsigned char)i;
    p.seg[NSEG] = np;
    for (int i = 0; i < NPANEL; ++i) p.pidx[p.panel[i]] = (unsigned char)i;
    for (int b = 0; b < NWG; ++b) {
        p.rec[b][0] = tile[b][0] | (unsigned)tile[b][1] << 16; p.rec[b][1] = tile[b][2] | (unsigned)tile[b][3] << 16;
        p.rec[b][2] = tile[b][4] | (unsigned)(role[b] << 9 | unit_of(p, ppos[b], 0)) << 16;
    }
}

}

extern "C" void kernel_launch(int rank, int nranks, ncclComm_t comm,
                              hipStream_t stream,
                              void* const* d_in, const int* in_sizes, int n_in,
                              void* d_out, int out_size,
                              const int* in_dtypes, int out_dtype,
                              void* const* d_out_all,
                              void* const* d_ws_all, size_t ws_bytes) {
    (void)comm; (void)in_dtypes; (void)out_dtype;
    if (nranks != NR || (unsigned)rank >= (unsigned)NR || n_in != 4 || in_sizes[0] != M_ * KSH || in_sizes[1] != KSH * N_ ||
        out_size != M_ * N_ || ws_bytes < WS_NEED) {
        fprintf(stderr, "kernel_launch: unexpected problem shape\n");
        return;
    }
    HIP_CHECK(hipFuncSetAttribute((const void*)fused_kernel, hipFuncAttributeMaxDynamicSharedMemorySize, LDS_BYTES));
    Params p;
    p.x = (const unsigned char*)d_in[0]; p.w = (const unsigned char*)d_in[1];
    p.sx = (const float*)d_in[2]; p.sw = (const float*)d_in[3];
    p.out = (float*)d_out;
    for (int i = 0; i < NR; ++i) { p.ws[i] = (unsigned char*)d_ws_all[i]; p.outs[i] = (const unsigned char*)d_out_all[i]; }
    p.rank = rank;
    p.mode = 0;
    make_schedule(p);
    hipLaunchKernelGGL(fused_kernel, dim3(NWG), dim3(NTHR), LDS_BYTES, stream, p);
    HIP_CHECK(hipGetLastError());
}
```

```cpp
#include <hip/hip_runtime.h>
#include <rccl/rccl.h>
#include <cstdio>

#define HIP_CHECK(expr) do { hipError_t _e = (expr); if (_e != hipSuccess) { \
    fprintf(stderr, "HIP error %s at %s:%d\n", hipGetErrorString(_e), __FILE__, __LINE__); \
    return; } } while (0)

namespace {

#define LAS __attribute__((address_space(3)))
typedef int i32x4 __attribute__((ext_vector_type(4)));
typedef int i32x8 __attribute__((ext_vector_type(8)));
typedef unsigned u32x4 __attribute__((ext_vector_type(4)));
typedef float f32x4 __attribute__((ext_vector_type(4)));
typedef float f32x2 __attribute__((ext_vector_type(2)));

constexpr int M_ = 4096, N_ = 16384, NR = 8, KSH = 512;
constexpr int NWG = 256, NTHR = 512;
constexpr unsigned UNIT = 32768;
constexpr unsigned HTB = 16384;
constexpr int NPANEL = 80, NU = 4 * NPANEL;
constexpr unsigned SRC_BYTES = NU * UNIT;
constexpr unsigned PAR_BYTES = NR * SRC_BYTES;
constexpr unsigned WS_CTR = 0;
constexpr unsigned WS_TMO = 16384;
constexpr unsigned WS_TK = 20480;
constexpr unsigned WS_LF = 32768;
constexpr unsigned WS_SF = 65536;
constexpr unsigned WS_DBG = 512 * 1024;
constexpr unsigned WS_STAGE = 1u << 20;
constexpr size_t WS_NEED = (size_t)WS_STAGE + 2ull * PAR_BYTES;
constexpr unsigned long long DEADLINE_TICKS = 40000000ull;

#define WPP 3
#define PULL_W 10
#define PULL_G 2
#define NSH 64
#define SH_INS 16
#define FIXSEG 3
#define PART0_W 8
constexpr int NPULL = 7 * WPP;
constexpr int MAXT = 5, NSEG = 6;
constexpr int SFN = NU + 16;
constexpr int NSO = NSH / NR, CSH = 64 - NSH / 16;
static_assert(NSH % 16 == 0 && NSO <= 16 && NU + NSH <= 464, "shared tiles");
constexpr int LDS_CTL = 8 * HTB, LDS_BYTES = LDS_CTL + 2048;
constexpr int ROLE_GEMM = 127, ROLE_PACK2 = 32;

struct Params {
    const unsigned char* x; const unsigned char* w; const float* sx; const float* sw;
    float* out;
    unsigned char* ws[NR];
    const unsigned char* outs[NR];
    int rank; int mode;
    int seg[NSEG + 1];
    unsigned char pbeg[NPANEL], plen[NPANEL];
    unsigned char panel[NPANEL];
    unsigned char pidx[NPANEL];
    unsigned rec[NWG][3];
};
static_assert(sizeof(Params) <= 4096, "kernel arguments");

__host__ __device__ __forceinline__ int unit_of(const Params& p, int o, int q) {
    int s0 = 0, n = 1;
#pragma unroll
    for (int s = 0; s < NSEG; ++s) if (o >= 4 * p.seg[s] && p.seg[s + 1] > p.seg[s]) { s0 = p.seg[s]; n = p.seg[s + 1] - s0; }
    o -= 4 * s0;
    const int j = (o >= n) + (o >= 2 * n) + (o >= 3 * n);
    int i = s0 + o - j * n;
    const int b = p.pbeg[i], m = p.plen[i];
    i += (q * m) / 7; if (i >= b + m) i -= m;
    return (int)p.panel[i] * 4 + j;
}

#define WAIT_V(n) asm volatile("s_waitcnt vmcnt(" #n ")" ::: "memory")
#define WAIT_VX(n) WAIT_V(n)
#define WAIT_L(n) asm volatile("s_waitcnt lgkmcnt(" #n ")" ::: "memory")
#define BAR __builtin_amdgcn_s_barrier()
#define FENCE asm volatile("" ::: "memory")
#define SCHED __builtin_amdgcn_sched_barrier(0)
#define STAMP(i) do { } while (0)

__device__ __forceinline__ void st16_sys(void* p, u32x4 v) { asm volatile("global_store_dwordx4 %0, %1, off sc0 sc1\n\ts_nop 1" :: "v"(p), "v"(v) : "memory"); }
__device__ __forceinline__ void ld4_dev_async(unsigned& v, const void* base, unsigned off) { asm volatile("global_load_dword %0, %1, %2 sc1" : "+v"(v) : "v"(off), "s"(base) : "memory"); }
__device__ __forceinline__ unsigned ld4_sys_now(const void* p) { unsigned v; asm volatile("global_load_dword %0, %1, off sc0 sc1\n\ts_waitcnt vmcnt(0)" : "=&v"(v) : "v"(p) : "memory"); return v; }
__device__ __forceinline__ unsigned ld4_dev_now(const void* p) { unsigned v; asm volatile("global_load_dword %0, %1, off sc1\n\ts_waitcnt vmcnt(0)" : "=&v"(v) : "v"(p) : "memory"); return v; }

__device__ __forceinline__ bool give_up(unsigned long long t0, unsigned call, unsigned* tmo) {
    if ((unsigned)__builtin_amdgcn_readfirstlane(ld4_dev_now(tmo)) == call) return true;
    if (__builtin_amdgcn_s_memrealtime() - t0 > DEADLINE_TICKS) {
        __hip_atomic_store(tmo, call, __ATOMIC_RELAXED, __HIP_MEMORY_SCOPE_AGENT);
        WAIT_V(0);
        return true;
    }
    return false;
}
__device__ __noinline__ void wait_flags(const unsigned* flag, unsigned call, unsigned* tmo) {
    const unsigned long long t0 = __builtin_amdgcn_s_memrealtime();
    for (unsigned it = 1;; ++it) {
        if (__all(ld4_dev_now(flag) >= call)) break;
        if ((it & 7u) == 0u && give_up(t0, call, tmo)) break;
        __builtin_amdgcn_s_sleep(12);
    }
}

__device__ __forceinline__ unsigned lf_index(int u, int src) {
    return (unsigned)(u >> 2) * 32u + (unsigned)(src * 4 + (u & 3));
}

__device__ __forceinline__ void pack_load(const Params& p, int u, u32x4 (&o)[4], int wid, int lane) {
    const int j = u & 3, pa = u >> 2;
    if (pa < 16) {
#pragma unroll
        for (int i = 0; i < 4; ++i) { const int pc = wid * 4 + i;
            const int row = 256 * pa + 16 * (pc >> 1) + (lane & 15), col = 128 * j + 64 * (pc & 1) + 16 * (lane >> 4);
            o[i] = *(const u32x4*)(p.x + (size_t)row * KSH + col); }
    } else {
        const unsigned char* src = p.w + (size_t)(128 * j + 16 * wid) * N_ + 256 * (pa - 16) + 4 * lane;
        unsigned r[16];
#pragma unroll
        for (int i = 0; i < 16; ++i) r[i] = *(const unsigned*)(src + (size_t)i * N_);
#pragma unroll
        for (int q = 0; q < 4; ++q) {
            const unsigned a = r[4 * q], b = r[4 * q + 1], c = r[4 * q + 2], d = r[4 * q + 3];
            const unsigned ab_lo = __builtin_amdgcn_perm(b, a, 0x05010400u), ab_hi = __builtin_amdgcn_perm(b, a, 0x07030602u);
            const unsigned cd_lo = __builtin_amdgcn_perm(d, c, 0x05010400u), cd_hi = __builtin_amdgcn_perm(d, c, 0x07030602u);
            o[0][q] = __builtin_amdgcn_perm(cd_lo, ab_lo, 0x05040100u);
            o[1][q] = __builtin_amdgcn_perm(cd_lo, ab_lo, 0x07060302u);
            o[2][q] = __builtin_amdgcn_perm(cd_hi, ab_hi, 0x05040100u);
            o[3][q] = __builtin_amdgcn_perm(cd_hi, ab_hi, 0x07060302u);
        }
    }
}
__device__ __forceinline__ void pack_store(int u, unsigned char* dst, const u32x4 (&o)[4], int wid, int lane) {
    const bool isx = (u >> 2) < 16;
    const unsigned base = isx ? (unsigned)wid * 4096u + (unsigned)lane * 16u
                              : (unsigned)(lane >> 4) * 4096u + (unsigned)(wid >> 2) * 1024u + (unsigned)((wid & 3) * 16 + (lane & 15)) * 16u;
#pragma unroll
    for (int c = 0; c < 4; ++c) st16_sys(dst + base + (isx ? c * 1024u : (c >> 1) * HTB + (c & 1) * 2048u), o[c]);
}

__device__ __forceinline__ float exp_clip(float t) {
    constexpr float LIM = 60.0f * 1.44269504088896340736f;
    return __builtin_amdgcn_exp2f(__builtin_amdgcn_fmed3f(t, -LIM, LIM));
}
__device__ __forceinline__ f32x4 silu4(float a0, float a1, float a2, float a3, float t0, float t1, float t2, float t3, float is) {
    const f32x2 e01 = {exp_clip(t0), exp_clip(t1)}, e23 = {exp_clip(t2), exp_clip(t3)};
    const f32x2 d01 = __builtin_elementwise_fma(e01, (f32x2){is, is}, (f32x2){is, is}), d23 = __builtin_elementwise_fma(e23, (f32x2){is, is}, (f32x2){is, is});
    float r0, r1, r2, r3;
    asm("v_rcp_f32 %0, %4\n\tv_rcp_f32 %1, %5\n\tv_rcp_f32 %2, %6\n\tv_rcp_f32 %3, %7\n\t"
        "v_mul_f32 %0, %8, %0\n\tv_mul_f32 %1, %9, %1\n\tv_mul_f32 %2, %10, %2\n\tv_mul_f32 %3, %11, %3"
        : "=&v"(r0), "=&v"(r1), "=&v"(r2), "=&v"(r3) : "v"(d01.x), "v"(d01.y), "v"(d23.x), "v"(d23.y), "v"(a0), "v"(a1), "v"(a2), "v"(a3));
    return (f32x4){r0, r1, r2, r3};
}

template <bool B> struct BoolC { static constexpr bool value = B; };

__global__ __launch_bounds__(NTHR, 2) void fused_kernel(const Params p) {
    extern __shared__ __attribute__((aligned(16))) unsigned char shm[];
    LAS unsigned char* lds = (LAS unsigned char*)shm;
    LAS unsigned* ctl = (LAS unsigned*)(lds + LDS_CTL);
    const int tid = threadIdx.x, wid = __builtin_amdgcn_readfirstlane(tid >> 6), lane = tid & 63;
    const int bid = blockIdx.x;
    const int rank = p.rank;
    unsigned char* ws = p.ws[rank];
    unsigned* tmo = (unsigned*)(ws + WS_TMO);
    STAMP(0);

    unsigned ticket = 0;
    if (tid == 0) ticket = __hip_atomic_fetch_add((unsigned*)(ws + WS_CTR + bid * 64), 1u, __ATOMIC_RELAXED, __HIP_MEMORY_SCOPE_AGENT);
    const unsigned rec0 = p.rec[bid][0], rec1 = p.rec[bid][1], rec2 = p.rec[bid][2];
    const int role = (int)(rec2 >> 25);
    int pu = (int)((rec2 >> 16) & 511u);
    u32x4 pk[4];
    if (role >= NPULL) pack_load(p, pu, pk, wid, lane);
    else if (tid < NU + NSH) {
        const int nin = 4 * p.seg[NSEG - 1];
        ((LAS unsigned short*)(ctl + 48))[tid] = (unsigned short)(tid < nin ? unit_of(p, tid, role % 7) : 512 + tid - nin);
    } else if (tid >= 464 + 2) ctl[tid - 464] = 0u;
    if (tid == 0) { ctl[0] = ticket + 1u; ctl[1] = 0u; }
    __syncthreads();
    const unsigned call = __builtin_amdgcn_readfirstlane(ctl[0]);
    const unsigned stage_off = WS_STAGE + (call & 1u) * PAR_BYTES;
    unsigned char* stage = ws + stage_off;

    if (!(p.mode & 1) && role >= NPULL) {
        for (int rep = 0;; ++rep) {
            pack_store(pu, stage + (size_t)rank * SRC_BYTES + (size_t)pu * UNIT, pk, wid, lane);
            WAIT_V(0); BAR; FENCE;
            if (tid < NR) {
                if (tid == rank) __hip_atomic_store((unsigned*)(ws + WS_LF) + lf_index(pu, rank), call, __ATOMIC_RELAXED, __HIP_MEMORY_SCOPE_AGENT);
                else __hip_atomic_store((unsigned*)(p.ws[tid] + WS_SF + (unsigned)(rank * SFN + pu) * 64u), call, __ATOMIC_RELAXED, __HIP_MEMORY_SCOPE_SYSTEM);
            }
            if (rep == 1 || role == ROLE_GEMM) break;
            pu = __builtin_amdgcn_readfirstlane(unit_of(p, NWG - NPULL + role - ROLE_PACK2, 0));
            pack_load(p, pu, pk, wid, lane);
        }
    }
    STAMP(1);

    if (role < NPULL) {
        if (p.mode & 2) return;
        const int q = role % 7, wq = role / 7, src = (rank + 1 + q) & (NR - 1);
        constexpr int C_PLAN = 2, C_ISSUED = 4, C_LANDED = 6, C_FREED = 8, C_DESC = 16, C_CNT = 32, C_TAB = 48;
        LAS unsigned short* utab = (LAS unsigned short*)(ctl + C_TAB);
        const unsigned cbase = (unsigned)(size_t)ctl;
#define CGET(w) ({ int v_; asm volatile("ds_read_b32 %0, %1\n\ts_waitcnt lgkmcnt(0)" : "=&v"(v_) : "v"(cbase + (unsigned)(w) * 4u) : "memory"); __builtin_amdgcn_readfirstlane(v_); })
#define CPUT(w, v) asm volatile("ds_write_b32 %0, %1" :: "v"(cbase + (unsigned)(w) * 4u), "v"((int)(v)) : "memory")
        const unsigned long long tstart = __builtin_amdgcn_s_memrealtime();
#define SPIN_GUARD() do { __builtin_amdgcn_s_sleep(4); if (__builtin_amdgcn_s_memrealtime() - tstart > 2 * DEADLINE_TICKS) { \
            __hip_atomic_store(tmo, call, __ATOMIC_RELAXED, __HIP_MEMORY_SCOPE_AGENT); return; } } while (0)
#define SH_OFF(z) ({ const int i_ = src * NSO + ((z) >> 3); ((size_t)((i_ & 15) * 256 + ((z) & 7) * 32) * N_ + (size_t)(CSH + (i_ >> 4)) * 256) * 4; })
        const int npos = 4 * p.seg[NSEG - 1] + NSH;
        if (wid < 2) {
            const unsigned char* rbase = p.ws[src] + stage_off + (size_t)src * SRC_BYTES + (unsigned)wid * HTB;
            const unsigned loff = (unsigned)lane * 16u;
            int landed = 0;
            for (int k = 0;; ++k) {
                for (int spin = 0;; ++spin) {
                    const int f0 = CGET(C_FREED + 2 * wid), f1 = CGET(C_FREED + 2 * wid + 1);
                    if (CGET(C_PLAN) > k && f0 + 4 > k && f1 + 4 > k) break;
                    if (landed < k && spin >= 16) { WAIT_V(0); landed = k; CPUT(C_LANDED + wid, k); } else SPIN_GUARD();
                }
                const int u = CGET(C_DESC + (k & 15));
                if (u < 0) { WAIT_V(0); CPUT(C_LANDED + wid, k); break; }
                const unsigned char* g = u < 512 ? rbase + (size_t)u * UNIT : p.outs[src] + SH_OFF(u - 512) + (size_t)wid * (16u * N_ * 4u);
                const unsigned gs = u < 512 ? 1024u : N_ * 4u;
                CPUT(C_ISSUED + wid, k + 1);
                static_assert(PULL_W % PULL_G == 0 && PULL_W <= 16 - PULL_G, "window");
#pragma unroll
                for (int i0 = 0; i0 < 16; i0 += PULL_G) {
                    WAIT_VX(PULL_W);
                    if (i0 == PULL_W && landed < k) { landed = k; CPUT(C_LANDED + wid, k); }
#pragma unroll
                    for (int i = i0; i < i0 + PULL_G; ++i)
                        __builtin_amdgcn_global_load_lds((const unsigned*)(g + i * gs + loff), (LAS unsigned*)(lds + (k & 3) * UNIT + wid * HTB + i * 1024), 16, 0, 17  );
                }
            }
            STAMP(2);
        } else if (wid == 2) {
            unsigned* tk = (unsigned*)(ws + WS_TK + q * 128);
            const unsigned tk_base = (call - 1u) * (unsigned)(npos - 2 * WPP);
            int horizon = 0;
            for (int k = 0;; ++k) {
                while (k > CGET(C_ISSUED) + 1 || k > CGET(C_ISSUED + 1) + 1) SPIN_GUARD();
                int pos;
                if (k < 3) pos = k * WPP + wq;
                else { unsigned t = 0; if (lane == 0) t = __hip_atomic_fetch_add(tk, 1u, __ATOMIC_RELAXED, __HIP_MEMORY_SCOPE_AGENT);
                    const unsigned r = (unsigned)__builtin_amdgcn_readfirstlane(t) - tk_base;
                    pos = r < (unsigned)(npos - 3 * WPP) ? 3 * WPP + (int)r : -1; }
                if (pos >= horizon) {
                    const unsigned long long t0 = __builtin_amdgcn_s_memrealtime();
                    horizon = pos;
                    for (;;) {
                        const int v = utab[horizon + lane < npos ? horizon + lane : npos - 1];
                        const unsigned long long nr = ~__ballot(ld4_sys_now(ws + WS_SF + (unsigned)(src * SFN + (v < 512 ? v : NU + ((v - 512) >> 3))) * 64u) >= call);
                        horizon += nr ? __builtin_ctzll(nr) : 64;
                        if (horizon > pos) break;
                        if (give_up(t0, call, tmo)) {
                            const int w = utab[pos];
                            if (w >= 512) { if (lane == 0) *(float*)((unsigned char*)p.out + SH_OFF(w - 512)) = __builtin_nanf(""); pos = -1; }
                            horizon = pos + 1; break; }
                        __builtin_amdgcn_s_sleep(2);
                    }
                }
                CPUT(C_DESC + (k & 15), pos >= 0 ? (int)utab[pos] : -1);
                CPUT(C_PLAN, k + 1);
                if (pos < 0) break;
            }
        } else if (wid >= 4) {
            const int i4 = wid - 4;
            const unsigned toff = (unsigned)i4 * 8192u + (unsigned)lane * 16u;
            unsigned char* lbase = stage + (size_t)src * SRC_BYTES + toff;
            int pend = -1, pend_u = 0;
#define ARRIVE() do { if (lane == 0) { unsigned old_; asm volatile("ds_add_rtn_u32 %0, %1, %2\n\ts_waitcnt lgkmcnt(0)" : "=&v"(old_) : "v"(cbase + (unsigned)(C_CNT + (pend & 15)) * 4u), "v"(1u) : "memory"); \
                if ((old_ & 3u) == 3u && pend_u < 512) { __hip_atomic_store((unsigned*)(ws + WS_LF) + lf_index(pend_u, src), call, __ATOMIC_RELAXED, __HIP_MEMORY_SCOPE_AGENT); PUBSTAMP(); } } } while (0)
#define PUBSTAMP() do { } while (0)
            for (int k = 0;; ++k) {
                bool end = false;
                while (CGET(C_LANDED + (i4 >> 1)) <= k) {
                    if (pend >= 0) { WAIT_V(0); ARRIVE(); pend = -1; }
                    else if (CGET(C_PLAN) > k && CGET(C_DESC + (k & 15)) < 0) { end = true; break; }
                    else SPIN_GUARD();
                }
                if (end) break;
                const int u = CGET(C_DESC + (k & 15));
                unsigned char* dst = u < 512 ? lbase + (size_t)u * UNIT : (unsigned char*)p.out + SH_OFF(u - 512) + (size_t)i4 * (8u * N_ * 4u) + lane * 16;
                const unsigned ds = u < 512 ? 1024u : N_ * 4u;
                const unsigned la = (unsigned)(size_t)(lds + (k & 3) * UNIT + toff);
#pragma unroll
                for (int h = 0; h < 2; ++h) { u32x4 t0, t1, t2, t3;
                    asm volatile("ds_read_b128 %0, %4\n\tds_read_b128 %1, %4 offset:1024\n\tds_read_b128 %2, %4 offset:2048\n\tds_read_b128 %3, %4 offset:3072\n\t"
                                 "s_waitcnt lgkmcnt(3)\n\tglobal_store_dwordx4 %5, %0, off sc1\n\t"
                                 "s_waitcnt lgkmcnt(2)\n\tglobal_store_dwordx4 %6, %1, off sc1\n\t"
                                 "s_waitcnt lgkmcnt(1)\n\tglobal_store_dwordx4 %7, %2, off sc1\n\t"
                                 "s_waitcnt lgkmcnt(0)\n\tglobal_store_dwordx4 %8, %3, off sc1\n\ts_nop 1"
                                 : "=&v"(t0), "=&v"(t1), "=&v"(t2), "=&v"(t3)
                                 : "v"(la + h * 4096), "v"(dst + (4 * h) * ds), "v"(dst + (4 * h + 1) * ds), "v"(dst + (4 * h + 2) * ds), "v"(dst + (4 * h + 3) * ds) : "memory"); }
                CPUT(C_FREED + i4, k + 1);
                if (pend >= 0) { WAIT_V(8); ARRIVE(); }
                pend = k; pend_u = u;
            }
        }
        return;
    }

    if (p.mode & 4) return;
    const int wr = wid >> 2, wc = wid & 3;
    int tdesc = -1;
    { const unsigned rw = lane < 2 ? rec0 : (lane < 4 ? rec1 : rec2), t = lane < MAXT ? (rw >> (16 * (lane & 1))) & 0xffffu : 0xffffu;
      if (t != 0xffffu) {
          const int ia = p.pidx[t >> 6], ib = p.pidx[16 + (t & 63u)], hi = ia > ib ? ia : ib, lo = ia > ib ? ib : ia;
          const int s0 = p.pbeg[hi], n = p.plen[hi];
          int key[7];
#pragma unroll
          for (int q = 0; q < 7; ++q) { const int r = (q * n) / 7; int kh = hi - s0 - r; if (kh < 0) kh += n; int kl = lo - s0 - r; if (kl < 0) kl += n;
              key[q] = (hi >= p.seg[FIXSEG] ? 0 : ((lo >= s0 && kl > kh) ? kl : kh)) * 8 + q; }
          unsigned w = 0;
#pragma unroll
          for (int q = 0; q < 7; ++q) { int rk = 0;
#pragma unroll
              for (int r = 0; r < 7; ++r) rk += key[r] < key[q];
              w |= (unsigned)q << (3 * rk); }
          tdesc = (int)(((t >> 6) << 27) | ((t & 63u) << 21) | w);
      } }
    const int ks = lane & 31, kj = ks < 4 ? ks : (ks - 4) / 7, ksh = ks < 4 ? 0 : 3 * ((ks - 4) - kj * 7);
#define KT_OF(d) (ks < 4 ? rank * 4 + ks : ((rank + 1 + (((d) >> ksh) & 7)) & 7) * 4 + kj)
#define PM_OF(d) (((d) >> 27) & 15)
#define PN_OF(d) (((d) >> 21) & 63)
#define FOFF_OF(d, kt) ((unsigned)(((lane < 32) ? PM_OF(d) : 16 + PN_OF(d)) * 32 + (kt)) * 4u)
    const unsigned* lf = (const unsigned*)(ws + WS_LF);
    const unsigned voff = (unsigned)tid * 16u;
    const unsigned ldsw = (unsigned)wid * 1024u;
    const LAS unsigned char* la = lds + wr * 8192 + lane * 16;
    const LAS unsigned char* lb = lds + 4 * HTB + wc * 4096 + lane * 16;
#define SA(b, h) (((b) * 2 + (h)) * HTB)
#define SB(b, h) ((4 + (b) * 2 + (h)) * HTB)
#define STAGE(bufoff, gbase) do { _Pragma("unroll") for (int _i = 0; _i < 2; ++_i) \
        __builtin_amdgcn_global_load_lds((const unsigned*)((gbase) + _i * 8192 + voff), (LAS unsigned*)(lds + (bufoff) + ldsw + _i * 8192), 16, 0, 16  ); } while (0)
#define LDFRAG(ptr) __builtin_shufflevector(*(const LAS i32x4*)(ptr), *(const LAS i32x4*)((ptr) + 1024), 0, 1, 2, 3, 4, 5, 6, 7)
#define LDA(dst, b, h) do { _Pragma("unroll") for (int m = 0; m < 4; ++m) dst[m] = LDFRAG(la + SA(b, h) + m * 2048); } while (0)
#define LDB(dst, b, h) do { _Pragma("unroll") for (int n = 0; n < 2; ++n) dst[n] = LDFRAG(lb + SA(b, h) + n * 2048); } while (0)
#define MFMA1(ai, bj, At, Bt, Z, i) do { constexpr int m = (i) >> 1, n = (i) & 1; \
        if constexpr (Z) asm volatile("v_mfma_f32_16x16x128_f8f6f4 %0, %1, %2, 0 cbsz:1 blgp:1" : "+v"(acc[ai][bj][m][n]) : "v"(At[m]), "v"(Bt[n])); \
        else acc[ai][bj][m][n] = __builtin_amdgcn_mfma_scale_f32_16x16x128_f8f6f4(At[m], Bt[n], acc[ai][bj][m][n], 1, 1, 0, 0, 0, 0); } while (0)
#define MMA2(ai, Z, B) do { __builtin_amdgcn_s_setprio(1); \
        MFMA1(ai, 0, At, B0, Z, 0); MFMA1(ai, 0, At, B0, Z, 1); MFMA1(ai, 1, At, B1, Z, 0); MFMA1(ai, 1, At, B1, Z, 1); \
        MFMA1(ai, 1, At, B1, Z, 3); MFMA1(ai, 1, At, B1, Z, 2); MFMA1(ai, 0, At, B0, Z, 3); MFMA1(ai, 0, At, B0, Z, 2); \
        MFMA1(ai, 0, At, B0, Z, 4); MFMA1(ai, 0, At, B0, Z, 5); MFMA1(ai, 1, At, B1, Z, 4); MFMA1(ai, 1, At, B1, Z, 5); \
        MFMA1(ai, 1, At, B1, Z, 7); MFMA1(ai, 1, At, B1, Z, 6); MFMA1(ai, 0, At, B0, Z, 7); MFMA1(ai, 0, At, B0, Z, 6); \
        __builtin_amdgcn_s_setprio(0); B; SCHED; } while (0)
#define KT(s) __builtin_amdgcn_readlane(kta, (s))
#define KOFF(kt) ((unsigned)((kt) >> 2) * SRC_BYTES + (unsigned)((kt) & 3) * UNIT)
#define MASK_OF(f) ({ const unsigned long long b_ = __ballot((f) >= call); (unsigned)b_ & (unsigned)(b_ >> 32); })
#define NEED(d, s, kt) do { if (!((mask >> (s)) & 1u)) wait_flags(lf + ((lane & 1) ? PM_OF(d) : 16 + PN_OF(d)) * 32 + (kt), call, tmo); } while (0)

    int ti = 0, cur = __builtin_amdgcn_readlane(tdesc, 0);
    if (cur < 0) return;
    f32x4 acc[2][2][4][2] = {};
    i32x8 At[4], B0[2], B1[2];
    const unsigned char* cA = stage + (size_t)PM_OF(cur) * 4 * UNIT;
    const unsigned char* cB = stage + (size_t)(16 + PN_OF(cur)) * 4 * UNIT;
    int kta = KT_OF(cur);
    unsigned foff = FOFF_OF(cur, kta), mask = 0, fb = 0, tv = 0;
    const float scale = __builtin_bit_cast(float, __builtin_amdgcn_readfirstlane(__builtin_bit_cast(int, p.sx[0] * p.sw[0])));
    const float sl = -scale * 1.44269504088896340736f, inv_s = 1.0f / scale;
    NEED(cur, 0, rank * 4); NEED(cur, 1, rank * 4 + 1);
    { const unsigned k0 = KOFF(rank * 4), k1 = k0 + UNIT;
      ld4_dev_async(fb, lf, foff);
      STAGE(SB(0, 0), cB + k0); STAGE(SA(0, 0), cA + k0); STAGE(SB(0, 1), cB + k0 + HTB); STAGE(SA(0, 1), cA + k0 + HTB);
      STAGE(SB(1, 0), cB + k1); STAGE(SA(1, 0), cA + k1); STAGE(SB(1, 1), cB + k1 + HTB);
      WAIT_V(0); BAR;
      if (wr == 1) BAR; }
    int pub = -1;
    bool rb = true;
    for (;;) {
        int nxt = __builtin_amdgcn_readlane(tdesc, ti + 1);
        const bool has_next = nxt >= 0;
        if (!has_next) nxt = cur;
        const unsigned char* nA = stage + (size_t)PM_OF(nxt) * 4 * UNIT;
        const unsigned char* nB = stage + (size_t)(16 + PN_OF(nxt)) * 4 * UNIT;
        auto trip = [&](auto first_c, const int t) __attribute__((always_inline)) {
            constexpr bool first = decltype(first_c)::value;
            const bool last = (t == 30);
            const int s2 = (t + 2) & 31, s3 = (t + 3) & 31;
            const int kt2 = KT(s2), kt3 = KT(s3);
            const unsigned k2 = KOFF(kt2), k3 = KOFF(kt3);
            const unsigned char* a1 = cA + KOFF(KT(t + 1));
            const unsigned char* a2 = (last ? nA : cA) + k2; const unsigned char* b2 = (last ? nB : cB) + k2;
            const unsigned char* a3 = (last ? nA : cA) + k3; const unsigned char* b3 = (last ? nB : cB) + k3;
            const int qd = last ? nxt : cur;
#define WAIT_S() do { if constexpr (first) WAIT_V(40); else WAIT_V(8); } while (0)
            if (last) ld4_dev_async(tv, tmo, 0u);
            LDB(B0, 0, 0); LDA(At, 0, 0); LDB(B1, 0, 1); STAGE(SA(1, 1), a1 + HTB);
            WAIT_S();
            if (rb) { asm volatile("" : "+v"(fb)); mask = MASK_OF(fb); }
            WAIT_L(0); BAR; MMA2(0, first, BAR);
            NEED(qd, s2, kt2);
            rb = (mask != 0xffffffffu);
            if (rb) ld4_dev_async(fb, lf, foff);
            LDA(At, 0, 1); STAGE(SB(0, 0), b2); STAGE(SA(0, 0), a2); STAGE(SB(0, 1), b2 + HTB);
            WAIT_S(); WAIT_L(0); BAR; MMA2(1, first, BAR);
            LDB(B0, 1, 0); LDA(At, 1, 0); LDB(B1, 1, 1); STAGE(SA(0, 1), a2 + HTB);
            WAIT_V(8);
            if constexpr (first) { if (pub >= 0) {
                unsigned old = 0;
                if (lane == 0) asm volatile("ds_add_rtn_u32 %0, %1, %2\n\ts_waitcnt lgkmcnt(0)" : "=&v"(old) : "v"((unsigned)(size_t)(ctl + 1)), "v"(1u) : "memory");
                if ((__builtin_amdgcn_readfirstlane(old) & 7) == 7 && lane < NR && lane != rank)
                    __hip_atomic_store((unsigned*)(p.ws[lane] + WS_SF + (unsigned)(rank * SFN + NU + pub) * 64u), call, __ATOMIC_RELAXED, __HIP_MEMORY_SCOPE_SYSTEM);
                pub = -1; } }
            if (rb) { asm volatile("" : "+v"(fb)); mask = MASK_OF(fb); }
            WAIT_L(0); BAR; MMA2(0, false, BAR);
            NEED(qd, s3, kt3);
            const bool sw = (t == 28);
            if (sw) foff = FOFF_OF(nxt, KT_OF(nxt));
            rb = sw || (mask != 0xffffffffu);
            if (rb) ld4_dev_async(fb, lf, foff);
            LDA(At, 1, 1); STAGE(SB(1, 0), b3); STAGE(SA(1, 0), a3); STAGE(SB(1, 1), b3 + HTB);
            WAIT_V(8);
            asm volatile("" : "+v"(tv));
            WAIT_L(0); BAR; MMA2(1, false, if (!(last && wr == 1)) BAR);
        };
        trip(BoolC<true>{}, 0);
        for (int t = 2; t < 32; t += 2) trip(BoolC<false>{}, t);
        STAMP(2 + 2 * ti);
        {
            const float is = ((unsigned)__builtin_amdgcn_readfirstlane(tv) == call) ? __builtin_nanf("") : inv_s;
            const float* ob = p.out + (size_t)(PM_OF(cur) * 256 + wr * 64) * N_ + PN_OF(cur) * 256 + wc * 64;
            const unsigned oo = (unsigned)(4 * (lane >> 4)) * (N_ * 4u) + (unsigned)(lane & 15) * 16u;
            auto epi = [&](auto shared_c, const unsigned oo) __attribute__((always_inline)) {
#pragma unroll
                for (int ai = 0; ai < 2; ++ai)
#pragma unroll
                    for (int m = 0; m < 4; ++m) {
                        const f32x4 a0 = acc[ai][0][m][0], a1 = acc[ai][0][m][1], a2 = acc[ai][1][m][0], a3 = acc[ai][1][m][1];
                        const f32x4 t0 = a0 * sl, t1 = a1 * sl, t2 = a2 * sl, t3 = a3 * sl;
#pragma unroll
                        for (int e = 0; e < 4; ++e) {
                            const f32x4 v = silu4(a0[e], a1[e], a2[e], a3[e], t0[e], t1[e], t2[e], t3[e], is);
                            const float* rowp = ob + (size_t)(ai * 128 + m * 16 + e) * N_;
                            if constexpr (decltype(shared_c)::value) asm volatile("global_store_dwordx4 %0, %1, %2 sc0 sc1\n\ts_nop 1" :: "v"(oo), "v"(v), "s"(rowp) : "memory");
                            else asm volatile("global_store_dwordx4 %0, %1, %2 nt\n\ts_nop 1" :: "v"(oo), "v"(v), "s"(rowp) : "memory"); } }
            };
            if (NSH > 0 && PN_OF(cur) >= CSH) pub = (PN_OF(cur) - CSH) * 16 + PM_OF(cur) - rank * NSO;
            if (pub >= 0) epi(BoolC<true>{}, oo);
            else epi(BoolC<false>{}, oo);
        }
        if (wr == 1) BAR;
        STAMP(3 + 2 * ti);
        if (!has_next) break;
        cur = nxt; cA = nA; cB = nB; kta = KT_OF(cur); ++ti;
    }
    WAIT_V(0);
    STAMP(14);
    if (wr == 0) BAR;
}

void make_schedule(Params& p) {
    constexpr int G = NWG - NPULL, E = 1024 - (NSH - NSO) - 4 * G;
    static_assert(E >= 0 && E <= G && E <= 256 && NPULL <= ROLE_PACK2 && ROLE_PACK2 + NU - G <= ROLE_GEMM, "schedule");
    int ngx[8] = {}, nex[8] = {};
    int role[NWG]; unsigned short tile[NWG][MAXT];
    int npx[8] = {}, ppos[NWG];
    for (int i = 0; i < NPULL; ++i) ++npx[2 * (i & 3) + 1];
    for (int b = 0; b < NWG; ++b) { role[b] = ROLE_GEMM; ppos[b] = 0; for (int i = 0; i < MAXT; ++i) tile[b][i] = 0xffff; }
    { int c = 0; for (int b = NWG - 1; b >= 0; --b) if ((b >> 3) >= 32 - npx[b & 7]) role[b] = c++; }
    for (int x = 0; x < 8; ++x) ngx[x] = 32 - npx[x];
    { int left = E; for (int pass = 0; pass < 2; ++pass) for (bool any = true; any && left > 0;) { any = false;
        for (int x = pass; x < 8 && left > 0; x += 2) if (nex[x] < ngx[x]) { ++nex[x]; --left; any = true; } } }
    { int c = 0; for (int b = 0; b < NWG; ++b) if (role[b] == ROLE_GEMM) ppos[b] = c++;
      c = 0; for (int b = NWG - 1; b >= 0 && c < NU - G; --b) if (role[b] == ROLE_GEMM) role[b] = ROLE_PACK2 + c++; }
    int sr = 0, sc = 0;
    if (E > 0) { int best = 1000; for (int a = 1; a <= 16; ++a) { const int b = (E + a - 1) / a; if (a + b <= best) { best = a + b; sr = a; sc = b; } } }
    int col[64];
    for (int c = 0; c < 64; ++c) col[c] = c < SH_INS ? c : (c < SH_INS + 64 - CSH ? CSH + c - SH_INS : c - (64 - CSH));
    signed char rnd[16][64]; int cnt = 0;
    for (int pn = 0; pn < 64; ++pn) for (int pm = 0; pm < 16; ++pm)
        rnd[pm][pn] = (pn >= CSH && ((pn - CSH) * 16 + pm) / (NSO ? NSO : 1) != p.rank) ? 99 : 100;
    for (int pn = 0; pn < sc; ++pn) for (int pm = 0; pm < sr; ++pm) if (cnt < E) { rnd[pm][pn] = -1; ++cnt; }
    cnt = 0;
    for (int c = 0; c < 64; ++c) for (int pm = 0; pm < 16; ++pm) if (rnd[pm][col[c]] == 100) { rnd[pm][col[c]] = (signed char)(cnt / G); ++cnt; }
    int ntile[NWG] = {};
    for (int r = -1; r < 4; ++r) {
        int xi = 0, slot = 0;
        for (int band = 0; band < 4; ++band)
            for (int c = 0; c < 64; ++c) { const int pn = col[c];
                for (int pm = 4 * band; pm < 4 * band + 4; ++pm) if (rnd[pm][pn] == r) {
                    while (slot >= (r < 0 ? nex[xi ^ 1] : ngx[xi ^ 1])) { ++xi; slot = 0; }
                    const int b = slot * 8 + (xi ^ 1); ++slot;
                    tile[b][ntile[b]++] = (unsigned short)(pm * 64 + pn);
                } }
    }
    bool used[NPANEL] = {}; int np = 0;
    for (int r = -1; r < 5; ++r) {
        p.seg[r + 1] = np;
        bool need[NPANEL] = {};
        for (int pm = 0; pm < 16; ++pm) for (int pn = 0; pn < 64; ++pn) if (rnd[pm][pn] == r || r == 4) { need[pm] = !used[pm]; need[16 + pn] = !used[16 + pn]; }
        int wl[64], nw = 0, nx = 0;
        for (int c = 0; c < 64; ++c) if (need[16 + col[c]]) wl[nw++] = 16 + col[c];
        for (int pm = 0; pm < 16; ++pm) nx += need[pm];
        const int nparts = (r < 0 || r == 4) ? 1 : 2;
        const int w0 = nparts == 1 ? nw : (nx ? PART0_W : nw / nparts);
        for (int part = 0, iw = 0; part < nparts; ++part) {
            const int beg = np, wend = part == 0 ? w0 : w0 + (nw - w0) * part / (nparts - 1);
            for (int ix = 0; iw < wend || (part == 0 && ix < 16);) {
                if (part == 0) { while (ix < 16 && !need[ix]) ++ix; if (ix < 16) { used[ix] = true; p.panel[np++] = (unsigned char)ix++; } }
                if (iw < wend) { used[wl[iw]] = true; p.panel[np++] = (unsigned char)wl[iw++]; }
            }
            for (int i = beg; i < np; ++i) { p.pbeg[i] = (unsigned char)beg; p.plen[i] = (unsigned char)(np - beg); }
        }
    }
    p.seg[NSEG] = np;
    for (int i = 0; i < NPANEL; ++i) p.pidx[p.panel[i]] = (unsigned char)i;
    for (int b = 0; b < NWG; ++b) {
        p.rec[b][0] = tile[b][0] | (unsigned)tile[b][1] << 16; p.rec[b][1] = tile[b][2] | (unsigned)tile[b][3] << 16;
        p.rec[b][2] = tile[b][4] | (unsigned)(role[b] << 9 | unit_of(p, ppos[b], 0)) << 16;
    }
}

}

extern "C" void kernel_launch(int rank, int nranks, ncclComm_t comm,
                              hipStream_t stream,
                              void* const* d_in, const int* in_sizes, int n_in,
                              void* d_out, int out_size,
                              const int* in_dtypes, int out_dtype,
                              void* const* d_out_all,
                              void* const* d_ws_all, size_t ws_bytes) {
    (void)comm; (void)in_dtypes; (void)out_dtype;
    if (nranks != NR || (unsigned)rank >= (unsigned)NR || n_in != 4 || in_sizes[0] != M_ * KSH || in_sizes[1] != KSH * N_ ||
        out_size != M_ * N_ || ws_bytes < WS_NEED) {
        fprintf(stderr, "kernel_launch: unexpected problem shape\n");
        return;
    }
    HIP_CHECK(hipFuncSetAttribute((const void*)fused_kernel, hipFuncAttributeMaxDynamicSharedMemorySize, LDS_BYTES));
    Params p;
    p.x = (const unsigned char*)d_in[0]; p.w = (const unsigned char*)d_in[1];
    p.sx = (const float*)d_in[2]; p.sw = (const float*)d_in[3];
    p.out = (float*)d_out;
    for (int i = 0; i < NR; ++i) { p.ws[i] = (unsigned char*)d_ws_all[i]; p.outs[i] = (const unsigned char*)d_out_all[i]; }
    p.rank = rank;
    p.mode = 0;
    make_schedule(p);
    hipLaunchKernelGGL(fused_kernel, dim3(NWG), dim3(NTHR), LDS_BYTES, stream, p);
    HIP_CHECK(hipGetLastError());
}
```
